```python
import numpy as np

kernel_name = "gpu_gemm_a2a_m4096_k4096_n16384_bf16_q8"


ABSMAX_RTOL = 0.02

COMPUTE_DTYPE = "bfloat16"
EPILOGUE = "quant_int8"
IN_QUANT = None
OUT_QUANT = "int8"
IN_DTYPES = {"x": "bfloat16", "w_mat": "bfloat16"}
WORLD_SIZE = 8

def _to_bf16(x):
    u = np.ascontiguousarray(x, dtype=np.float32).view(np.uint32)
    return ((u + 0x7FFF + ((u >> 16) & 1)) & 0xFFFF0000).view(np.float32)

def setup_inputs():

    x = _to_bf16(np.random.randn(4096, 4096).astype(np.float32))
    w_mat = _to_bf16((np.random.randn(4096, 16384) / np.sqrt(4096)).astype(np.float32))
    return {"x": x, "w_mat": w_mat}

def _quant_dequant(y, amax):
    scale = amax / 127.0
    q = np.clip(np.round(y / scale), -127.0, 127.0)
    return (q * scale).astype(np.float32)
def _epilogue(y):
    return _quant_dequant(y, np.abs(y).max())

def reference(x, w_mat):
    y = (x @ w_mat).astype(np.float32)
    return _epilogue(y)
```

```cpp
#include <hip/hip_runtime.h>
#include <rccl/rccl.h>
#include <cstdio>
#include <cstdint>

#define HIP_CHECK(expr) do { hipError_t _e = (expr); if (_e != hipSuccess) { \
    fprintf(stderr, "HIP error %s at %s:%d\n", hipGetErrorString(_e), __FILE__, __LINE__); \
    return; } } while (0)

namespace {
#define LAS __attribute__((address_space(3)))
typedef short bf16x8 __attribute__((ext_vector_type(8)));
typedef short s16x4 __attribute__((ext_vector_type(4)));
typedef float f32x4 __attribute__((ext_vector_type(4)));
typedef unsigned u32x4 __attribute__((ext_vector_type(4)));
typedef unsigned u32x2 __attribute__((ext_vector_type(2)));

constexpr int NR = 8;
constexpr int MLOC = 512, KD = 4096, ND = 16384, NSH = ND / NR;
constexpr int BM = 256, BN = 128, BK = 64, NT = KD / BK;
constexpr int NWG = 256;
constexpr int A_BYTES = BM * BK * 2, B_BYTES = BK * BN * 2, SLOT = A_BYTES + B_BYTES, NSLOT = 3;
constexpr int LDS_BYTES = NSLOT * SLOT;
constexpr size_t WS_CNT = 0;
constexpr size_t WS_TMO = 4096;
constexpr size_t WS_AMAX = 8192;
constexpr size_t WS_DONE = 65536;
constexpr size_t WS_STAMP = 131072;
constexpr unsigned long long DEADLINE = 50000000ull;

struct Params {
  const unsigned short* x; const unsigned short* w;
  char* out[NR]; char* ws[NR];
  int rank; int nsync;
};

#define WAIT_V(n) asm volatile("s_waitcnt vmcnt(" #n ")" ::: "memory")
#define BAR() do { asm volatile("" ::: "memory"); __builtin_amdgcn_s_barrier(); asm volatile("" ::: "memory"); } while (0)
#define LGKM0() __builtin_amdgcn_s_waitcnt(0xC07F)

#define STAMP(i) do {} while (0)

__device__ __forceinline__ void store16_sys(void* ptr, u32x4 v) {
  asm volatile("global_store_dwordx4 %0, %1, off sc0 sc1\n\ts_nop 1" :: "v"(ptr), "v"(v) : "memory");
}
__device__ __forceinline__ float wave_max(float v) {
#pragma unroll
  for (int o = 32; o >= 1; o >>= 1) v = fmaxf(v, __shfl_xor(v, o));
  return v;
}

__global__ __launch_bounds__(512, 2) void gemm_a2a_q8(const Params p) {
  __shared__ __attribute__((aligned(16))) char smem[LDS_BYTES];
  LAS char* lds = (LAS char*)smem;
  const int tid = threadIdx.x, lane = tid & 63;
  const int wid = __builtin_amdgcn_readfirstlane(tid >> 6);
  const int b = blockIdx.x, xg = b & 7, j = b >> 3;
  const int dst = (xg + p.rank) & 7;
  const int rt = j & 1, ct = j >> 1;
  const int n0 = dst * NSH + ct * BN, m0 = rt * BM;
  char* const myws = p.ws[p.rank];
  LAS float* red = (LAS float*)lds;

  if (wid >= 4) {
    const int lw = wid - 4;
    if (wid == 4) STAMP(0);
    const char* ga[2]; const char* gb[2];
#pragma unroll
    for (int h = 0; h < 2; ++h) {
      const int row = 64 * lw + 8 * h + (lane >> 3);
      const int c = (lane & 7) ^ ((row >> 1) & 7);
      ga[h] = (const char*)p.x + (size_t)(m0 + row) * (KD * 2) + c * 16;
      const int kr = 16 * lw + 8 * h + (lane >> 3);
      const int ch = (lane & 7) ^ ((h << 2) | ((lane >> 3) & 2));
      gb[h] = (const char*)p.w + (size_t)kr * (ND * 2) + (size_t)n0 * 2 + ch * 16;
    }
    auto issue = [&](int t, int slot) {
      LAS char* sa = lds + slot * SLOT + lw * 8192;
      LAS char* sb = lds + slot * SLOT + A_BYTES + lw * 4096;
#pragma unroll
      for (int i = 0; i < 8; ++i)
        __builtin_amdgcn_global_load_lds((const unsigned*)(ga[i & 1] + (size_t)(i >> 1) * (16 * KD * 2) + (size_t)t * (BK * 2)),
                                         (LAS unsigned*)(sa + i * 1024), 16, 0, 0);
#pragma unroll
      for (int i = 0; i < 4; ++i)
        __builtin_amdgcn_global_load_lds((const unsigned*)(gb[i >> 1] + (size_t)(i & 1) * 128 + (size_t)t * ((size_t)BK * ND * 2)),
                                         (LAS unsigned*)(sb + i * 1024), 16, 0, 0);
    };
    issue(0, 0); issue(1, 1); issue(2, 2);
    WAIT_V(24); BAR();
    int slot = 0;
    for (int t = 1; t < NT - 2; ++t) {
      WAIT_V(12); BAR();
      issue(t + 2, slot);
      slot = (slot == NSLOT - 1) ? 0 : slot + 1;
    }
    WAIT_V(12); BAR();
    WAIT_V(0); BAR();
    unsigned call = 0;
    if (wid == 4) {
      unsigned* cnt = (unsigned*)(myws + WS_CNT) + b;
      call = __hip_atomic_load(cnt, __ATOMIC_RELAXED, __HIP_MEMORY_SCOPE_AGENT) + 1;
      if (lane == 0) __hip_atomic_store(cnt, call, __ATOMIC_RELAXED, __HIP_MEMORY_SCOPE_AGENT);
    } else {
    }
    BAR();
    BAR();
    if (wid == 4) {
      STAMP(1);
      const float m = fmaxf(fmaxf(red[0], red[1]), fmaxf(red[2], red[3]));
      if (lane < p.nsync) {
        const int pr = (p.nsync == 1) ? p.rank : lane;
        unsigned long long* g = (unsigned long long*)(p.ws[pr] + WS_AMAX) + ((size_t)(call & 1) * NR + p.rank) * NWG + b;
        __hip_atomic_store(g, ((unsigned long long)call << 32) | __float_as_uint(m), __ATOMIC_RELAXED, __HIP_MEMORY_SCOPE_SYSTEM);
      }
      if (lane == 0) ((LAS unsigned*)red)[16] = call;
      LGKM0();
    }
    BAR();
    call = ((LAS unsigned*)red)[16];
    {
      unsigned* tmo = (unsigned*)(myws + WS_TMO);
      const char* gp = myws + WS_AMAX + ((size_t)(call & 1) * NR + 2 * lw) * NWG * 8 + lane * 16;
      const bool use0 = (p.nsync == NR) || (2 * lw == p.rank), use1 = (p.nsync == NR) || (2 * lw + 1 == p.rank);
      const unsigned long long t0 = __builtin_amdgcn_s_memrealtime();
      float m = 0.f; bool bad = false;
      for (;;) {
        u32x4 a, c, d, e;
        asm volatile("global_load_dwordx4 %0, %4, off sc0 sc1\n\t"
                     "global_load_dwordx4 %1, %4, off offset:1024 sc0 sc1\n\t"
                     "global_load_dwordx4 %2, %4, off offset:2048 sc0 sc1\n\t"
                     "global_load_dwordx4 %3, %4, off offset:3072 sc0 sc1\n\t"
                     "s_waitcnt vmcnt(0)"
                     : "=&v"(a), "=&v"(c), "=&v"(d), "=&v"(e) : "v"(gp) : "memory");
        const bool ok0 = a.y == call && a.w == call && c.y == call && c.w == call;
        const bool ok1 = d.y == call && d.w == call && e.y == call && e.w == call;
        const bool ok = (ok0 || !use0) && (ok1 || !use1);
        if (__all(ok)) {
          const unsigned m0u = max(max(a.x, a.z), max(c.x, c.z)), m1u = max(max(d.x, d.z), max(e.x, e.z));
          m = __uint_as_float(max(use0 ? m0u : 0u, use1 ? m1u : 0u));
          break;
        }
        if (__hip_atomic_load(tmo, __ATOMIC_RELAXED, __HIP_MEMORY_SCOPE_AGENT) == call ||
            __builtin_amdgcn_s_memrealtime() - t0 > DEADLINE) {
          __hip_atomic_store(tmo, call, __ATOMIC_RELAXED, __HIP_MEMORY_SCOPE_AGENT);
          bad = true; break;
        }
        __builtin_amdgcn_s_sleep(1);
      }
      m = wave_max(m);
      if (__hip_atomic_load(tmo, __ATOMIC_RELAXED, __HIP_MEMORY_SCOPE_AGENT) == call) bad = true;
      if (lane == 0) { red[4 + lw] = m; ((LAS unsigned*)red)[8 + lw] = bad ? 1u : 0u; }
      LGKM0();
    }
    if (wid == 4) STAMP(2);
    BAR();
    BAR();
    if (wid == 4) {
      STAMP(3);
      if (lane == 0 && dst != p.rank && p.nsync == NR)
        __hip_atomic_store((unsigned*)(p.ws[dst] + WS_DONE) + p.rank * 32 + j, call, __ATOMIC_RELAXED, __HIP_MEMORY_SCOPE_SYSTEM);
      if (b == 0 && p.nsync == NR) {
        unsigned* tmo = (unsigned*)(myws + WS_TMO);
        const char* fp = myws + WS_DONE + lane * 16;
        const bool mine = (lane >> 3) == p.rank;
        const unsigned long long t0 = __builtin_amdgcn_s_memrealtime();
        for (;;) {
          u32x4 a;
          asm volatile("global_load_dwordx4 %0, %1, off sc0 sc1\n\ts_waitcnt vmcnt(0)" : "=&v"(a) : "v"(fp) : "memory");
          const bool ok = mine || (a.x >= call && a.y >= call && a.z >= call && a.w >= call);
          if (__all(ok)) break;
          if (__hip_atomic_load(tmo, __ATOMIC_RELAXED, __HIP_MEMORY_SCOPE_AGENT) == call ||
              __builtin_amdgcn_s_memrealtime() - t0 > DEADLINE) {
            __hip_atomic_store(tmo, call, __ATOMIC_RELAXED, __HIP_MEMORY_SCOPE_AGENT);
            if (lane == 0) p.out[p.rank][(size_t)(p.rank * MLOC) * NSH] = (char)-128;
            break;
          }
          __builtin_amdgcn_s_sleep(1);
        }
      }
      STAMP(4);
    }
    return;
  }

  const int wm = wid >> 1, wn = wid & 1, fr = lane & 15, fq = lane >> 4;
  f32x4 acc[8][4];
#pragma unroll
  for (int m = 0; m < 8; ++m)
#pragma unroll
    for (int n = 0; n < 4; ++n) acc[m][n] = (f32x4){0.f, 0.f, 0.f, 0.f};
  int offA[2], offB[4];
#pragma unroll
  for (int ks = 0; ks < 2; ++ks) offA[ks] = (wm * 128 + fr) * 128 + (((4 * ks + fq) ^ ((fr >> 1) & 7)) << 4);
  {
    const int q = (lane & 15) >> 2, pp = lane & 3;
#pragma unroll
    for (int n = 0; n < 4; ++n)
      offB[n] = A_BYTES + 2048 * fq + 1024 * wn + 128 * q + (((2 * n + (pp >> 1)) ^ (((fq & 1) << 2) | (q & 2))) << 4) + 8 * (pp & 1);
  }
  bf16x8 Af[2][8], Bf[2][4];
#define LOADF(buf, ks) do { \
    _Pragma("unroll") for (int n = 0; n < 4; ++n) { \
      const s16x4 lo = __builtin_amdgcn_ds_read_tr16_b64_v4i16((LAS s16x4*)(lds + offB[n] + (ks) * 8192)); \
      const s16x4 hi = __builtin_amdgcn_ds_read_tr16_b64_v4i16((LAS s16x4*)(lds + offB[n] + (ks) * 8192 + 512)); \
      Bf[buf][n] = __builtin_shufflevector(lo, hi, 0, 1, 2, 3, 4, 5, 6, 7); } \
    _Pragma("unroll") for (int m = 0; m < 8; ++m) Af[buf][m] = *(const LAS bf16x8*)(lds + offA[ks] + m * 2048); \
  } while (0)
#define MMA(buf) do { \
    _Pragma("unroll") for (int m = 0; m < 8; ++m) \
      _Pragma("unroll") for (int n = 0; n < 4; ++n) \
        acc[m][n] = __builtin_amdgcn_mfma_f32_16x16x32_bf16(Bf[buf][n], Af[buf][m], acc[m][n], 0, 0, 0); \
  } while (0)
#define INTERLEAVE() do { \
    _Pragma("unroll") for (int i = 0; i < 16; ++i) { \
      __builtin_amdgcn_sched_group_barrier(0x008, 1, 0); \
      __builtin_amdgcn_sched_group_barrier(0x100, 1, 0); } \
    __builtin_amdgcn_sched_group_barrier(0x008, 16, 0); \
  } while (0)

  BAR();
  LOADF(0, 0);
  int slot = 0;
  for (int t = 0; t < NT; ++t) {
    LOADF(1, 1); MMA(0); INTERLEAVE();
    __builtin_amdgcn_sched_barrier(0);
    const int d = (slot == NSLOT - 1) ? -(NSLOT - 1) * SLOT : SLOT;
    slot = (slot == NSLOT - 1) ? 0 : slot + 1;
    offA[0] += d; offA[1] += d; offB[0] += d; offB[1] += d; offB[2] += d; offB[3] += d;
    LGKM0(); BAR();
    __builtin_amdgcn_sched_barrier(0);
    LOADF(0, 0); MMA(1); INTERLEAVE();
    __builtin_amdgcn_sched_barrier(0);
  }

  float mx = 0.f;
#pragma unroll
  for (int m = 0; m < 8; ++m)
#pragma unroll
    for (int n = 0; n < 4; ++n) {
      mx = fmaxf(mx, fmaxf(fabsf(acc[m][n][0]), fabsf(acc[m][n][1])));
      mx = fmaxf(mx, fmaxf(fabsf(acc[m][n][2]), fabsf(acc[m][n][3])));
    }
  mx = wave_max(mx);
  if (lane == 0) red[wid] = mx;
  LGKM0(); BAR();
  BAR();
  BAR();
  const float amax = fmaxf(fmaxf(red[4], red[5]), fmaxf(red[6], red[7]));
  const LAS unsigned* badw = (const LAS unsigned*)red + 8;
  const bool bad = (badw[0] | badw[1] | badw[2] | badw[3]) != 0u;
  const float inv = 127.0f / amax;
  const float MAGIC = 12582912.0f;
  char* orow = p.out[dst] + (size_t)(p.rank * MLOC + m0 + wm * 128 + fr) * NSH + ct * BN + wn * 64 + fq * 16;
#pragma unroll
  for (int m = 0; m < 8; ++m) {
    unsigned dw[4];
#pragma unroll
    for (int n = 0; n < 4; ++n) {
      unsigned q[4];
#pragma unroll
      for (int e = 0; e < 4; ++e)
        q[e] = __float_as_uint(__builtin_amdgcn_fmed3f(__builtin_fmaf(acc[m][n][e], inv, MAGIC), MAGIC - 127.0f, MAGIC + 127.0f));
      const unsigned lo = __builtin_amdgcn_perm(q[1], q[0], 0x0c0c0400u), hi = __builtin_amdgcn_perm(q[3], q[2], 0x04000c0cu);
      dw[n] = lo | hi;
    }
    auto s02 = __builtin_amdgcn_permlane32_swap(dw[0], dw[2], false, false);
    auto s13 = __builtin_amdgcn_permlane32_swap(dw[1], dw[3], false, false);
    auto s01 = __builtin_amdgcn_permlane16_swap(s02[0], s13[0], false, false);
    auto s23 = __builtin_amdgcn_permlane16_swap(s02[1], s13[1], false, false);
    u32x4 v = {s01[0], s01[1], s23[0], s23[1]};
    if (bad) v = (u32x4){0x80808080u, 0x80808080u, 0x80808080u, 0x80808080u};
    store16_sys(orow + (size_t)m * 16 * NSH, v);
  }
  WAIT_V(0);
  BAR();
}
}

extern "C" void kernel_launch(int rank, int nranks, ncclComm_t comm, hipStream_t stream,
                              void* const* d_in, const int* in_sizes, int n_in,
                              void* d_out, int out_size, const int* in_dtypes, int out_dtype,
                              void* const* d_out_all, void* const* d_ws_all, size_t ws_bytes) {
  if ((nranks != NR && nranks != 1) || (unsigned)rank >= (unsigned)NR || ws_bytes < (1u << 20)) { fprintf(stderr, "unsupported configuration\n"); return; }
  Params p;
  p.x = (const unsigned short*)d_in[0]; p.w = (const unsigned short*)d_in[1];
  for (int i = 0; i < NR; ++i) { p.out[i] = (char*)d_out_all[i]; p.ws[i] = (char*)d_ws_all[i]; }
  p.rank = rank; p.nsync = nranks;
  hipLaunchKernelGGL(gemm_a2a_q8, dim3(NWG), dim3(512), 0, stream, p);
  HIP_CHECK(hipGetLastError());
}
```

```cpp
#include <hip/hip_runtime.h>
#include <rccl/rccl.h>
#include <cstdio>
#include <cstdint>

#define HIP_CHECK(expr) do { hipError_t _e = (expr); if (_e != hipSuccess) { \
    fprintf(stderr, "HIP error %s at %s:%d\n", hipGetErrorString(_e), __FILE__, __LINE__); \
    return; } } while (0)

#define SRCSLOT p.rank
#define SOLO_ 0
#define KSTAG (xg * 8)
#define PSLEEP0 12
#define PSLEEP 6
#define PARX 1
#define FAUX ""
#define WAUX ""
namespace {
#define LAS __attribute__((address_space(3)))
typedef short bf16x8 __attribute__((ext_vector_type(8)));
typedef short s16x4 __attribute__((ext_vector_type(4)));
typedef float f32x4 __attribute__((ext_vector_type(4)));
typedef unsigned u32x4 __attribute__((ext_vector_type(4)));

constexpr int NR = 8;
constexpr int MLOC = 512, KD = 4096, ND = 16384, NSH = ND / NR;
constexpr int BM = 256, BN = 128, BK = 64, NT = KD / BK;
constexpr int NWG = 256;
constexpr int A_BYTES = BM * BK * 2, H_BYTES = BK * BN, NAS = 3, NLS = 5, NFS = 3;
constexpr int LDS_A = 0, LDS_L = NAS * A_BYTES, LDS_F = LDS_L + NLS * H_BYTES, LDS_BYTES = LDS_F + NFS * H_BYTES;
constexpr int LDS_CTL = LDS_A + A_BYTES;
constexpr int C_RED = 0;
constexpr int C_BAD = 8;
constexpr size_t WS_CNT = 0;
constexpr size_t WS_TMO = 4096;
constexpr size_t WS_AMAX = 8192;
constexpr size_t WS_DONE = 65536;
constexpr size_t WS_STAMP = 262144;
constexpr size_t WS_NEED = 1u << 20;
constexpr unsigned long long DEADLINE = 50000000ull;

struct Params {
  const unsigned short* x; const unsigned short* w;
  char* out[NR]; char* ws[NR];
  int rank; int pad;
};

#define WAIT_V(n) asm volatile("s_waitcnt vmcnt(%0)" :: "i"(n) : "memory")
#define BAR() do { asm volatile("" ::: "memory"); __builtin_amdgcn_s_barrier(); asm volatile("" ::: "memory"); } while (0)
#define LGKM0() __builtin_amdgcn_s_waitcnt(0xC07F)
#define RFL(x) __builtin_amdgcn_readfirstlane(x)
#define SB() __builtin_amdgcn_sched_barrier(0)

#define STAMP(i) do {} while (0)

__device__ __forceinline__ void store16_sys(void* ptr, u32x4 v) {
  asm volatile("global_store_dwordx4 %0, %1, off sc0 sc1\n\ts_nop 1" :: "v"(ptr), "v"(v) : "memory");
}
__device__ __forceinline__ unsigned wave_umax(unsigned v) {
  v = max(v, (unsigned)__builtin_amdgcn_update_dpp(0, (int)v, 0xB1, 0xF, 0xF, true));
  v = max(v, (unsigned)__builtin_amdgcn_update_dpp(0, (int)v, 0x4E, 0xF, 0xF, true));
  v = max(v, (unsigned)__builtin_amdgcn_update_dpp(0, (int)v, 0x141, 0xF, 0xF, true));
  v = max(v, (unsigned)__builtin_amdgcn_update_dpp(0, (int)v, 0x140, 0xF, 0xF, true));
  auto a = __builtin_amdgcn_permlane16_swap(v, v, false, false); v = max(a[0], a[1]);
  auto c = __builtin_amdgcn_permlane32_swap(v, v, false, false); return max(c[0], c[1]);
}
#define GL4(base, v, dst, aux) do { unsigned keep_; \
    asm volatile("s_mov_b32 %0, m0\n\ts_mov_b32 m0, %6\n\ts_nop 0\n\t" \
                 "global_load_lds_dwordx4 %1, %5" aux "\n\t" \
                 "global_load_lds_dwordx4 %2, %5 offset:1024" aux "\n\t" \
                 "global_load_lds_dwordx4 %3, %5 offset:2048" aux "\n\t" \
                 "global_load_lds_dwordx4 %4, %5 offset:3072" aux "\n\t" \
                 "s_mov_b32 m0, %0" \
                 : "=&s"(keep_) : "v"((v)[0]), "v"((v)[1]), "v"((v)[2]), "v"((v)[3]), "s"(base), "s"(dst) : "memory"); } while (0)

__global__ __launch_bounds__(320, 2) void gemm_a2a_q8(const Params p) {
  __shared__ __attribute__((aligned(16))) char smem[LDS_BYTES];
  LAS char* lds = (LAS char*)smem;
  LAS unsigned* ctl = (LAS unsigned*)(lds + LDS_CTL);
  const int tid = threadIdx.x, lane = tid & 63;
  const int wid = RFL(tid >> 6);
  const int b = blockIdx.x, xg = b & 7, j = b >> 3;
  const int dst = (xg + p.rank) & 7;
  const int ct = j >> 1, m0 = (j & 1) * BM;
  const int n0 = dst * NSH + ct * BN, rt = j & 1;
  const int kt0 = KSTAG;
  char* const myws = p.ws[p.rank];

  unsigned* const cnt = (unsigned*)(myws + WS_CNT) + b;
  unsigned* const tmo = (unsigned*)(myws + WS_TMO);

  if (wid == 4) {
    STAMP(0);
    const unsigned ldsb = (unsigned)(uintptr_t)lds;
    unsigned va[4], vb[4];
#pragma unroll
    for (int i = 0; i < 4; ++i) {
      const int row = 8 * i + (lane >> 3);
      va[i] = (unsigned)(row * (KD * 2) + (((lane & 7) ^ ((row >> 1) & 7)) << 4)) + 4096u - i * 1024u;
      const int kr = 8 * (i >> 1) + (lane >> 3);
      vb[i] = (unsigned)(kr * (ND * 2) + (i & 1) * 128 + (((lane & 7) ^ (((i >> 1) << 2) | ((lane >> 3) & 2))) << 4)) + 4096u - i * 1024u;
    }
    const char* abase = (const char*)p.x + (size_t)m0 * (KD * 2) - 4096;
    const char* lbase = (const char*)p.w + (size_t)n0 * 2 + (size_t)rt * (32 * ND * 2) - 4096;
    const char* fbase = (const char*)p.w + (size_t)n0 * 2 + (size_t)(1 - rt) * (32 * ND * 2) - 4096;
    auto issue_ah = [&](int t, int slot, int h) {
      const char* at = abase + ((t + kt0) & (NT - 1)) * (BK * 2) + (size_t)h * (128 * KD * 2);
#pragma unroll
      for (int g = 0; g < 4; ++g) GL4(at + (size_t)g * (32 * KD * 2), va, ldsb + LDS_A + slot * A_BYTES + h * 16384 + g * 4096, "");
    };
    auto issue_a = [&](int t, int slot) { issue_ah(t, slot, 0); issue_ah(t, slot, 1); };
    auto issue_l = [&](int t, int slot) {
      const char* bt = lbase + (unsigned)(((t + kt0) & (NT - 1)) * (BK * ND * 2));
      GL4(bt, vb, ldsb + LDS_L + slot * H_BYTES, WAUX); GL4(bt + 16 * ND * 2, vb, ldsb + LDS_L + slot * H_BYTES + 4096, WAUX);
    };
    auto issue_f = [&](int t, int slot) {
      const char* bt = fbase + (unsigned)(((t + kt0) & (NT - 1)) * (BK * ND * 2));
      GL4(bt, vb, ldsb + LDS_F + slot * H_BYTES, FAUX); GL4(bt + 16 * ND * 2, vb, ldsb + LDS_F + slot * H_BYTES + 4096, FAUX);
    };
    const int par = (int)(((unsigned)((uintptr_t)p.x >> 7) + kt0 + PARX) & 1);
    issue_l(0, 0); issue_f(0, 0); issue_a(0, 0); issue_l(1, 1); issue_f(1, 1);
    WAIT_V(16); BAR();
    issue_a(1, 1); issue_l(2, 2); issue_f(2, 2); issue_ah(2, 2, 0);
    WAIT_V(32); BAR();
    issue_ah(2, 2, 1); issue_l(3, 3);
    if (par == 0) issue_l(4, 4);
    int sa = 0, sl = par ? 4 : 0, tl = par ? 4 : 5;
    int prev_l = par ? 8 : 16;
    for (int t = 1; t < NT - 1; ++t) {
      int n = 0, nl = 0;
      if (t + 2 < NT) { issue_a(t + 2, sa); issue_f(t + 2, sa); sa = sa == 2 ? 0 : sa + 1; n = 40; }
      if ((t & 1) == par) {
#pragma unroll
        for (int i = 0; i < 2; ++i)
          if (tl < NT) { issue_l(tl, sl); ++tl; sl = sl == NLS - 1 ? 0 : sl + 1; nl += 8; }
      }
      n += nl + prev_l; prev_l = nl;
      if (n >= 56) WAIT_V(56); else if (n >= 40) WAIT_V(40); else WAIT_V(0);
      BAR();
    }
    const unsigned call = RFL(__hip_atomic_load(cnt, __ATOMIC_RELAXED, __HIP_MEMORY_SCOPE_AGENT)) + 1;
    unsigned long long* gran = (unsigned long long*)(p.ws[lane & 7] + WS_AMAX) + ((size_t)(call & 1) * NR + SRCSLOT) * NWG + b;
    asm volatile("" : "+v"(gran));
    BAR();
    const unsigned m = max(max(ctl[C_RED], ctl[C_RED + 1]), max(ctl[C_RED + 2], ctl[C_RED + 3]));
    if (lane < NR) __hip_atomic_store(gran, ((unsigned long long)call << 32) | m, __ATOMIC_RELAXED, __HIP_MEMORY_SCOPE_SYSTEM);
    if (lane == 0) __hip_atomic_store(cnt, call, __ATOMIC_RELAXED, __HIP_MEMORY_SCOPE_AGENT);
    STAMP(1);
    BAR();
    STAMP(2);
    BAR();
    if (lane == 0 && dst != p.rank)
      __hip_atomic_store((unsigned*)(p.ws[dst] + WS_DONE) + p.rank * 32 + j, call, __ATOMIC_RELAXED, __HIP_MEMORY_SCOPE_SYSTEM);
    STAMP(3);
    if (b == 0 && !SOLO_) {
      const char* fp = myws + WS_DONE + lane * 16;
      const bool mine = (lane >> 3) == p.rank;
      const unsigned long long t0 = __builtin_amdgcn_s_memrealtime();
      for (;;) {
        u32x4 a; unsigned tw;
        asm volatile("global_load_dwordx4 %0, %2, off sc0 sc1\n\tglobal_load_dword %1, %3, off sc1\n\ts_waitcnt vmcnt(0)" : "=&v"(a), "=&v"(tw) : "v"(fp), "v"(tmo) : "memory");
        const bool ok = mine || (a.x >= call && a.y >= call && a.z >= call && a.w >= call);
        if (__all(ok)) break;
        if (RFL(tw) == call || __builtin_amdgcn_s_memrealtime() - t0 > DEADLINE) {
          __hip_atomic_store(tmo, call, __ATOMIC_RELAXED, __HIP_MEMORY_SCOPE_AGENT);
          if (lane == 0) p.out[p.rank][(size_t)(p.rank * MLOC) * NSH] = (char)-128;
          break;
        }
        __builtin_amdgcn_s_sleep(1);
      }
    }
    STAMP(4);
    return;
  }

  const int wm = wid >> 1, wn = wid & 1, fr = lane & 15, fq = lane >> 4;
  const unsigned call = RFL(__hip_atomic_load(cnt, __ATOMIC_RELAXED, __HIP_MEMORY_SCOPE_AGENT)) + 1;
  f32x4 acc[8][4];
#pragma unroll
  for (int m = 0; m < 8; ++m)
#pragma unroll
    for (int n = 0; n < 4; ++n) acc[m][n] = (f32x4){0.f, 0.f, 0.f, 0.f};
  int offA[2], offAh[2], offB[4];
#pragma unroll
  for (int ks = 0; ks < 2; ++ks) { offA[ks] = LDS_A + (wm * 128 + fr) * 128 + (((4 * ks + fq) ^ ((fr >> 1) & 7)) << 4); offAh[ks] = offA[ks] + 2 * A_BYTES; }
  {
    const int q = (lane & 15) >> 2, pp = lane & 3;
#pragma unroll
    for (int n = 0; n < 4; ++n)
      offB[n] = (rt == 0 ? LDS_L : LDS_F) + 2048 * fq + 1024 * wn + 128 * q + (((2 * n + (pp >> 1)) ^ (((fq & 1) << 2) | (q & 2))) << 4) + 8 * (pp & 1);
  }
  asm volatile("" : "+v"(offAh[0]), "+v"(offAh[1]));
  bf16x8 Af[2][8];
  s16x4 Bl[2][4], Bh[2][4];
#define RD(buf, sl, ks, i) do { \
    if ((i) < 8) { \
      const s16x4 v_ = __builtin_amdgcn_ds_read_tr16_b64_v4i16((LAS s16x4*)(lds + offB[((i) >> 1) & 3] + ((i) & 1) * 512)); \
      if ((i) & 1) Bh[buf][((i) >> 1) & 3] = v_; else Bl[buf][((i) >> 1) & 3] = v_; \
    } else Af[buf][(i) & 7] = *(const LAS bf16x8*)(lds + ((sl) == 2 ? offAh[ks] : offA[ks] + (sl) * A_BYTES) + ((i) & 7) * 2048); \
  } while (0)
#define MF(buf, i) asm volatile("v_mfma_f32_16x16x32_bf16 %0, %1, %2, %0" : "+v"(acc[(i) >> 2][(i) & 3]) \
    : "v"(__builtin_shufflevector(Bl[buf][(i) & 3], Bh[buf][(i) & 3], 0, 1, 2, 3, 4, 5, 6, 7)), "v"(Af[buf][(i) >> 2]))
#define MMA(buf, i0, i1) do { _Pragma("unroll") for (int i = (i0); i < (i1); ++i) { MF(buf, i); SB(); } } while (0)
#define MMA_RD(buf, sl, ks) do { _Pragma("unroll") for (int i = 0; i < 16; ++i) { MF(buf, i); RD(1 - (buf), sl, ks, i); SB(); } } while (0)
#define LPOS 31
  const int rb0 = rt == 0 ? LDS_L : LDS_F, rb1 = rt == 0 ? LDS_F : LDS_L;
  const int re0 = rb0 + (rt == 0 ? NLS : NFS) * H_BYTES, re1 = rb1 + (rt == 0 ? NFS : NLS) * H_BYTES;
  int p0 = rb0, p1 = rb1;
#define MOVB(buf, d) do { _Pragma("unroll") for (int i = 0; i < 4; ++i) { MF(buf, 16 + i); offB[i] += (d); asm volatile("" : "+v"(offB[i])); SB(); } } while (0)
#define TILE(sl, nx) do { \
    MMA_RD(0, sl, 1); \
    p0 += H_BYTES; p0 = p0 == re0 ? rb0 : p0; \
    MOVB(0, p0 - p1); \
    MMA(0, 20, LPOS); \
    LGKM0(); BAR(); SB(); \
    MMA(0, LPOS, 32); \
    MMA_RD(1, nx, 0); \
    p1 += H_BYTES; p1 = p1 == re1 ? rb1 : p1; \
    MOVB(1, p1 - p0); \
    MMA(1, 20, 32); \
  } while (0)

  BAR();
#pragma unroll
  for (int i = 0; i < 16; ++i) RD(0, 0, 0, i);
  SB();
#pragma unroll
  for (int i = 0; i < 4; ++i) offB[i] += p1 - p0;
  SB();
  for (int t = 0; t < NT - 1; t += 3) {
    TILE(0, 1);
    TILE(1, 2);
    TILE(2, 0);
  }
  MMA_RD(0, 0, 1); MMA(0, 16, 32);
  float mx0 = 0.f, mx1 = 0.f;
#define AMX(i) do { \
    asm volatile("v_max3_f32 %0, |%1|, |%2|, %0" : "+v"(mx0) : "v"(acc[(i) >> 2][(i) & 3][0]), "v"(acc[(i) >> 2][(i) & 3][1])); \
    asm volatile("v_max3_f32 %0, |%1|, |%2|, %0" : "+v"(mx1) : "v"(acc[(i) >> 2][(i) & 3][2]), "v"(acc[(i) >> 2][(i) & 3][3])); } while (0)
  MMA(1, 0, 8);
#pragma unroll
  for (int i = 8; i < 32; ++i) { MF(1, i); AMX(i - 8); SB(); }
#pragma unroll
  for (int i = 24; i < 32; ++i) { AMX(i); SB(); }
  if (wid == 0) STAMP(7);
  const unsigned mw = wave_umax(max(__float_as_uint(mx0), __float_as_uint(mx1)));
  if (lane == 0) ctl[C_RED + wid] = mw;
  char* orow = p.out[dst] + (size_t)(p.rank * MLOC + m0 + wm * 128 + fr) * NSH + ct * BN + wn * 64 + fq * 16;
  LGKM0(); BAR();
  {
    const char* gp = myws + WS_AMAX + ((size_t)(call & 1) * NR + 2 * wid) * NWG * 8 + lane * 16;
    const unsigned long long t0 = __builtin_amdgcn_s_memrealtime();
    unsigned m = 0; bool gave_up = false;
    __builtin_amdgcn_s_sleep(PSLEEP0);
    for (;;) {
      u32x4 a, c, d, e; unsigned tw;
      asm volatile("global_load_dwordx4 %0, %5, off sc0 sc1\n\t"
                   "global_load_dwordx4 %1, %5, off offset:1024 sc0 sc1\n\t"
                   "global_load_dwordx4 %2, %5, off offset:2048 sc0 sc1\n\t"
                   "global_load_dwordx4 %3, %5, off offset:3072 sc0 sc1\n\t"
                   "global_load_dword %4, %6, off sc1\n\t"
                   "s_waitcnt vmcnt(0)"
                   : "=&v"(a), "=&v"(c), "=&v"(d), "=&v"(e), "=&v"(tw) : "v"(gp), "v"(tmo) : "memory");
      const bool ok = a.y == call && a.w == call && c.y == call && c.w == call && d.y == call && d.w == call && e.y == call && e.w == call;
      if (RFL(tw) == call) { gave_up = true; break; }
      if (__all(ok)) { m = max(max(max(a.x, a.z), max(c.x, c.z)), max(max(d.x, d.z), max(e.x, e.z))); break; }
      if (__builtin_amdgcn_s_memrealtime() - t0 > DEADLINE) { __hip_atomic_store(tmo, call, __ATOMIC_RELAXED, __HIP_MEMORY_SCOPE_AGENT); gave_up = true; break; }
      __builtin_amdgcn_s_sleep(PSLEEP);
    }
    m = wave_umax(m);
    if (lane == 0) { ctl[C_RED + 4 + wid] = m; ctl[C_BAD + wid] = gave_up ? 1u : 0u; }
  }
  LGKM0(); BAR();
  const float amax = __uint_as_float(max(max(ctl[C_RED + 4], ctl[C_RED + 5]), max(ctl[C_RED + 6], ctl[C_RED + 7])));
  const bool bad = (ctl[C_BAD] | ctl[C_BAD + 1] | ctl[C_BAD + 2] | ctl[C_BAD + 3]) != 0u;
  const float inv = 127.0f / amax;
  const float MAGIC = 12582912.0f;
#pragma unroll
  for (int m = 0; m < 8; ++m) {
    unsigned dw[4];
#pragma unroll
    for (int n = 0; n < 4; ++n) {
      unsigned q[4];
#pragma unroll
      for (int e = 0; e < 4; ++e)
        q[e] = __float_as_uint(__builtin_amdgcn_fmed3f(__builtin_fmaf(acc[m][n][e], inv, MAGIC), MAGIC - 127.0f, MAGIC + 127.0f));
      const unsigned lo = __builtin_amdgcn_perm(q[1], q[0], 0x0c0c0400u), hi = __builtin_amdgcn_perm(q[3], q[2], 0x04000c0cu);
      dw[n] = lo | hi;
    }
    auto s02 = __builtin_amdgcn_permlane32_swap(dw[0], dw[2], false, false);
    auto s13 = __builtin_amdgcn_permlane32_swap(dw[1], dw[3], false, false);
    auto s01 = __builtin_amdgcn_permlane16_swap(s02[0], s13[0], false, false);
    auto s23 = __builtin_amdgcn_permlane16_swap(s02[1], s13[1], false, false);
    u32x4 v = {s01[0], s01[1], s23[0], s23[1]};
    if (bad) v = (u32x4){0x80808080u, 0x80808080u, 0x80808080u, 0x80808080u};
    store16_sys(orow + (size_t)m * 16 * NSH, v);
    SB();
  }
  WAIT_V(0);
  BAR();
}
}

extern "C" void kernel_launch(int rank, int nranks, ncclComm_t comm, hipStream_t stream,
                              void* const* d_in, const int* in_sizes, int n_in,
                              void* d_out, int out_size, const int* in_dtypes, int out_dtype,
                              void* const* d_out_all, void* const* d_ws_all, size_t ws_bytes) {
  if ((nranks != NR && !SOLO_) || (unsigned)rank >= (unsigned)NR || ws_bytes < WS_NEED) { fprintf(stderr, "unsupported configuration\n"); return; }
  Params p;
  p.x = (const unsigned short*)d_in[0]; p.w = (const unsigned short*)d_in[1];
  for (int i = 0; i < NR; ++i) { p.out[i] = (char*)d_out_all[i]; p.ws[i] = (char*)d_ws_all[i]; }
  p.rank = rank; p.pad = 0;
  hipLaunchKernelGGL(gemm_a2a_q8, dim3(NWG), dim3(320), 0, stream, p);
  HIP_CHECK(hipGetLastError());
}
```

```cpp
#include <hip/hip_runtime.h>
#include <rccl/rccl.h>
#include <cstdio>
#include <cstdint>

#define HIP_CHECK(expr) do { hipError_t _e = (expr); if (_e != hipSuccess) { \
    fprintf(stderr, "HIP error %s at %s:%d\n", hipGetErrorString(_e), __FILE__, __LINE__); \
    return; } } while (0)

#define SRCSLOT p.rank
#define SOLO_ 0
#define KSTAG (xg * 8)
#define PSLEEP0 12
#define PSLEEP 6
#define PARX 1
#define FAUX ""
#define WAUX ""
namespace {
#define LAS __attribute__((address_space(3)))
typedef short bf16x8 __attribute__((ext_vector_type(8)));
typedef short s16x4 __attribute__((ext_vector_type(4)));
typedef float f32x4 __attribute__((ext_vector_type(4)));
typedef unsigned u32x4 __attribute__((ext_vector_type(4)));

constexpr int NR = 8;
constexpr int MLOC = 512, KD = 4096, ND = 16384, NSH = ND / NR;
constexpr int BM = 256, BN = 128, BK = 64, NT = KD / BK;
constexpr int NWG = 256;
constexpr int A_BYTES = BM * BK * 2, H_BYTES = BK * BN, NAS = 3, NLS = 5, NFS = 3;
constexpr int LDS_A = 0, LDS_L = NAS * A_BYTES, LDS_F = LDS_L + NLS * H_BYTES, LDS_BYTES = LDS_F + NFS * H_BYTES;
constexpr int LDS_CTL = LDS_A + A_BYTES;
constexpr int LDS_XCH = LDS_A + 2 * A_BYTES;
constexpr int C_RED = 0;
constexpr int C_BAD = 8;
constexpr size_t WS_CNT = 0;
constexpr size_t WS_TMO = 4096;
constexpr size_t WS_AMAX = 8192;
constexpr size_t WS_DONE = 65536;
constexpr size_t WS_STAMP = 262144;
constexpr size_t WS_NEED = 1u << 20;
constexpr unsigned long long DEADLINE = 50000000ull;

struct Params {
  const unsigned short* x; const unsigned short* w;
  char* out[NR]; char* ws[NR];
  int rank; int pad;
};

#define WAIT_V(n) asm volatile("s_waitcnt vmcnt(%0)" :: "i"(n) : "memory")
#define BAR() do { asm volatile("" ::: "memory"); __builtin_amdgcn_s_barrier(); asm volatile("" ::: "memory"); } while (0)
#define LGKM0() __builtin_amdgcn_s_waitcnt(0xC07F)
#define RFL(x) __builtin_amdgcn_readfirstlane(x)
#define SB() __builtin_amdgcn_sched_barrier(0)

#define STAMP(i) do {} while (0)
#define STAMP5()

__device__ __forceinline__ void store16_sys(void* ptr, u32x4 v) {
  asm volatile("global_store_dwordx4 %0, %1, off sc0 sc1\n\ts_nop 1" :: "v"(ptr), "v"(v) : "memory");
}
__device__ __forceinline__ unsigned wave_umax(unsigned v) {
  v = max(v, (unsigned)__builtin_amdgcn_update_dpp(0, (int)v, 0xB1, 0xF, 0xF, true));
  v = max(v, (unsigned)__builtin_amdgcn_update_dpp(0, (int)v, 0x4E, 0xF, 0xF, true));
  v = max(v, (unsigned)__builtin_amdgcn_update_dpp(0, (int)v, 0x141, 0xF, 0xF, true));
  v = max(v, (unsigned)__builtin_amdgcn_update_dpp(0, (int)v, 0x140, 0xF, 0xF, true));
  auto a = __builtin_amdgcn_permlane16_swap(v, v, false, false); v = max(a[0], a[1]);
  auto c = __builtin_amdgcn_permlane32_swap(v, v, false, false); return max(c[0], c[1]);
}
#define G8(aux) \
    "global_load_lds_dwordx4 %2, %0 offset:-4096" aux "\n\tglobal_load_lds_dwordx4 %3, %0 offset:-3072" aux "\n\t" \
    "global_load_lds_dwordx4 %4, %0 offset:-2048" aux "\n\tglobal_load_lds_dwordx4 %5, %0 offset:-1024" aux "\n\t" \
    "global_load_lds_dwordx4 %6, %0" aux "\n\tglobal_load_lds_dwordx4 %7, %0 offset:1024" aux "\n\t" \
    "global_load_lds_dwordx4 %8, %0 offset:2048" aux "\n\tglobal_load_lds_dwordx4 %9, %0 offset:3072" aux "\n\t"
#define GL8(base, v, dst, aux) \
    asm volatile("s_mov_b32 m0, %1\n\ts_nop 0\n\t" G8(aux) \
                 :: "s"(base), "s"((dst) + 4096), "v"((v)[0]), "v"((v)[1]), "v"((v)[2]), "v"((v)[3]), "v"((v)[4]), "v"((v)[5]), "v"((v)[6]), "v"((v)[7]) : "memory")

__global__ __launch_bounds__(320, 2) void gemm_a2a_q8(const Params p) {
  __shared__ __attribute__((aligned(16))) char smem[LDS_BYTES];
  LAS char* lds = (LAS char*)smem;
  LAS unsigned* ctl = (LAS unsigned*)(lds + LDS_CTL);
  const int tid = threadIdx.x, lane = tid & 63;
  const int wid = RFL(tid >> 6);
  const int b = blockIdx.x, xg = b & 7, j = b >> 3;
  const int dst = (xg + p.rank) & 7;
  const int ct = j >> 1, m0 = (j & 1) * BM;
  const int n0 = dst * NSH + ct * BN, rt = j & 1;
  const int kt0 = KSTAG;
  char* const myws = p.ws[p.rank];

  unsigned* const cnt = (unsigned*)(myws + WS_CNT) + b;
  unsigned* const tmo = (unsigned*)(myws + WS_TMO);
#define LOAD_CALL(call) unsigned call; asm volatile("s_load_dword %0, %1, 0x0\n\ts_waitcnt lgkmcnt(0)" : "=s"(call) : "s"(cnt) : "memory"); call += 1

  if (wid == 4) {
    STAMP(0);
    const unsigned ldsb = (unsigned)(uintptr_t)lds;
    unsigned va[8], vb[8];
#pragma unroll
    for (int i = 0; i < 8; ++i) {
      const int kr = 8 * (i >> 1) + (lane >> 3);
      vb[i] = (unsigned)(kr * (ND * 2) + (i & 1) * 128 + (((lane & 7) ^ ((((i >> 1) & 1) << 2) | ((lane >> 3) & 2))) << 4)) + 8192u - i * 1024u;
      const int row = 8 * i + (lane >> 3);
      va[i] = (unsigned)(row * (KD * 2) + (((lane & 7) ^ ((row >> 1) & 7)) << 4)) + 8192u - i * 1024u;
    }
    const char* abase = (const char*)p.x + (size_t)m0 * (KD * 2) - 4096;
    const char* lbase = (const char*)p.w + (size_t)n0 * 2 + (size_t)rt * (32 * ND * 2) - 4096;
    const char* fbase = (const char*)p.w + (size_t)n0 * 2 + (size_t)(1 - rt) * (32 * ND * 2) - 4096;
    auto issue_ah = [&](int t, int slot, int h) {
      const char* at = abase + ((t + kt0) & (NT - 1)) * (BK * 2) + (size_t)h * (128 * KD * 2);
#pragma unroll
      for (int g = 0; g < 2; ++g) GL8(at + (size_t)g * (64 * KD * 2), va, ldsb + LDS_A + slot * A_BYTES + h * 16384 + g * 8192, "");
    };
    auto issue_a = [&](int t, int slot) { issue_ah(t, slot, 0); issue_ah(t, slot, 1); };
    auto issue_l = [&](int t, int slot) { GL8(lbase + (unsigned)(((t + kt0) & (NT - 1)) * (BK * ND * 2)), vb, ldsb + LDS_L + slot * H_BYTES, WAUX); };
    auto issue_f = [&](int t, int slot) { GL8(fbase + (unsigned)(((t + kt0) & (NT - 1)) * (BK * ND * 2)), vb, ldsb + LDS_F + slot * H_BYTES, FAUX); };
    const int par = (int)(((unsigned)((uintptr_t)p.x >> 7) + kt0 + PARX) & 1);
    issue_l(0, 0); issue_f(0, 0); issue_a(0, 0); issue_l(1, 1); issue_f(1, 1);
    LOAD_CALL(call);
    char* pw = p.ws[0];
#pragma unroll
    for (int i = 1; i < NR; ++i) pw = (lane & 7) == i ? p.ws[i] : pw;
    unsigned long long* gran = (unsigned long long*)(pw + WS_AMAX) + ((size_t)(call & 1) * NR + SRCSLOT) * NWG + b;
    asm volatile("" : "+v"(gran));
    WAIT_V(16); BAR();
    STAMP5();
    issue_a(1, 1); issue_l(2, 2); issue_f(2, 2); issue_ah(2, 2, 0);
    WAIT_V(32); BAR();
    issue_ah(2, 2, 1); issue_l(3, 3);
    if (par == 0) issue_l(4, 4);
    int sa = 0, sl = par ? 4 : 0, tl = par ? 4 : 5;
    int prev_l = par ? 8 : 16;
#define XPF 0
#define DEFER 0
    const unsigned vx = (unsigned)((16 * ct + (lane >> 3)) * (KD * 2) + (lane & 7) * 16) + 4096u;
    auto xpf = [&](int t, int slot) {
      const char* a0 = abase + ((t + kt0) & (NT - 1)) * (BK * 2); const char* a1 = abase + ((t + 1 + kt0) & (NT - 1)) * (BK * 2);
      asm volatile("s_mov_b32 m0, %4\n\ts_nop 0\n\tglobal_load_lds_dwordx4 %0, %2\n\tglobal_load_lds_dwordx4 %1, %2\n\tglobal_load_lds_dwordx4 %0, %3\n\tglobal_load_lds_dwordx4 %1, %3"
                   :: "v"(vx), "v"(vx + 8 * KD * 2), "s"(a0), "s"(a1), "s"(ldsb + LDS_L + slot * H_BYTES) : "memory");
    };
    for (int t = 1; t < NT - 1; ++t) {
      int n = 0, nl = 0;
      const bool lead = (t & 1) == par;
      if (DEFER) {
        const int sp = sa == 0 ? 2 : sa - 1;
        if (!lead && t > 1) { issue_f(t + 1, sp); prev_l = 0; }
        if (t + 2 < NT) { issue_a(t + 2, sa); n = 32; if (!lead) { issue_f(t + 2, sa); n = 40; } }
        sa = sa == 2 ? 0 : sa + 1;
      } else if (t + 2 < NT) { issue_a(t + 2, sa); issue_f(t + 2, sa); sa = sa == 2 ? 0 : sa + 1; n = 40; }
      if (lead) {
        if (XPF && tl < NT) { xpf(t + XPF, sl); nl += 4; }
#pragma unroll
        for (int i = 0; i < 2; ++i)
          if (tl < NT) { issue_l(tl, sl); ++tl; sl = sl == NLS - 1 ? 0 : sl + 1; nl += 8; }
      }
      n += nl + prev_l; prev_l = nl;
      if (n >= 56) WAIT_V(56); else if (n >= 48) WAIT_V(48); else if (n >= 40) WAIT_V(40); else if (n >= 32) WAIT_V(32); else WAIT_V(0);
      BAR();
    }
    BAR();
    const unsigned m = max(max(ctl[C_RED], ctl[C_RED + 1]), max(ctl[C_RED + 2], ctl[C_RED + 3]));
    if (lane < NR) __hip_atomic_store(gran, ((unsigned long long)call << 32) | m, __ATOMIC_RELAXED, __HIP_MEMORY_SCOPE_SYSTEM);
    if (lane == 0) __hip_atomic_store(cnt, call, __ATOMIC_RELAXED, __HIP_MEMORY_SCOPE_AGENT);
    STAMP(1);
    BAR();
    STAMP(2);
    BAR(); BAR();
    BAR();
    if (lane == 0 && dst != p.rank)
      __hip_atomic_store((unsigned*)(p.ws[dst] + WS_DONE) + p.rank * 32 + j, call, __ATOMIC_RELAXED, __HIP_MEMORY_SCOPE_SYSTEM);
    STAMP(3);
    if (b == 0 && !SOLO_) {
      const char* fp = myws + WS_DONE + lane * 16;
      const bool mine = (lane >> 3) == p.rank;
      const unsigned long long t0 = __builtin_amdgcn_s_memrealtime();
      for (;;) {
        u32x4 a; unsigned tw;
        asm volatile("global_load_dwordx4 %0, %2, off sc0 sc1\n\tglobal_load_dword %1, %3, off sc1\n\ts_waitcnt vmcnt(0)" : "=&v"(a), "=&v"(tw) : "v"(fp), "v"(tmo) : "memory");
        const bool ok = mine || (a.x >= call && a.y >= call && a.z >= call && a.w >= call);
        if (__all(ok)) break;
        if (RFL(tw) == call || __builtin_amdgcn_s_memrealtime() - t0 > DEADLINE) {
          __hip_atomic_store(tmo, call, __ATOMIC_RELAXED, __HIP_MEMORY_SCOPE_AGENT);
          if (lane == 0) p.out[p.rank][(size_t)(p.rank * MLOC) * NSH] = (char)-128;
          break;
        }
        __builtin_amdgcn_s_sleep(1);
      }
    }
    STAMP(4);
    return;
  }

  const int wm = wid >> 1, wn = wid & 1, fr = lane & 15, fq = lane >> 4;
  LOAD_CALL(call);
  f32x4 acc[8][4];
#pragma unroll
  for (int m = 0; m < 8; ++m)
#pragma unroll
    for (int n = 0; n < 4; ++n) acc[m][n] = (f32x4){0.f, 0.f, 0.f, 0.f};
  int offA[2], offAh[2], offB[4];
#pragma unroll
  for (int ks = 0; ks < 2; ++ks) { offA[ks] = LDS_A + (wm * 128 + fr) * 128 + (((4 * ks + fq) ^ ((fr >> 1) & 7)) << 4); offAh[ks] = offA[ks] + 2 * A_BYTES; }
  {
    const int q = (lane & 15) >> 2, pp = lane & 3;
#pragma unroll
    for (int n = 0; n < 4; ++n)
      offB[n] = (rt == 0 ? LDS_L : LDS_F) + 2048 * fq + 1024 * wn + 128 * q + (((2 * n + (pp >> 1)) ^ (((fq & 1) << 2) | (q & 2))) << 4) + 8 * (pp & 1);
  }
  asm volatile("" : "+v"(offAh[0]), "+v"(offAh[1]));
  bf16x8 Af[2][8];
  s16x4 Bl[2][4], Bh[2][4];
#define RD(buf, sl, ks, i) do { \
    if ((i) < 8) { \
      const s16x4 v_ = __builtin_amdgcn_ds_read_tr16_b64_v4i16((LAS s16x4*)(lds + offB[((i) >> 1) & 3] + ((i) & 1) * 512)); \
      if ((i) & 1) Bh[buf][((i) >> 1) & 3] = v_; else Bl[buf][((i) >> 1) & 3] = v_; \
    } else Af[buf][(i) & 7] = *(const LAS bf16x8*)(lds + ((sl) == 2 ? offAh[ks] : offA[ks] + (sl) * A_BYTES) + ((i) & 7) * 2048); \
  } while (0)
#define ORD 1
#define MI(i) ((i) >> 2)
#define NI(i) ((((i) >> 2) & 1) ? 3 - ((i) & 3) : ((i) & 3))
#define MFOPS "%0, %1, %2, %0"
#define MF(buf, i) asm volatile("v_mfma_f32_16x16x32_bf16 " MFOPS : "+v"(acc[MI(i)][NI(i)]) \
    : "v"(__builtin_shufflevector(Bl[buf][NI(i)], Bh[buf][NI(i)], 0, 1, 2, 3, 4, 5, 6, 7)), "v"(Af[buf][MI(i)]))
#define MMA(buf, i0, i1) do { _Pragma("unroll") for (int i = (i0); i < (i1); ++i) { MF(buf, i); SB(); } } while (0)
#define MMA_RD(buf, sl, ks) do { _Pragma("unroll") for (int i = 0; i < 16; ++i) { MF(buf, i); RD(1 - (buf), sl, ks, i); SB(); } } while (0)
#define LPOS 31
  const int rb0 = rt == 0 ? LDS_L : LDS_F, rb1 = rt == 0 ? LDS_F : LDS_L;
  const int re0 = rb0 + (rt == 0 ? NLS : NFS) * H_BYTES, re1 = rb1 + (rt == 0 ? NFS : NLS) * H_BYTES;
  int p0 = rb0, p1 = rb1;
#define MOVB(buf, d) do { _Pragma("unroll") for (int i = 0; i < 4; ++i) { MF(buf, 16 + i); offB[i] += (d); asm volatile("" : "+v"(offB[i])); SB(); } } while (0)
#define TILE(sl, nx) do { \
    MMA_RD(0, sl, 1); \
    p0 += H_BYTES; p0 = p0 == re0 ? rb0 : p0; \
    MOVB(0, p0 - p1); \
    MMA(0, 20, LPOS); \
    LGKM0(); BAR(); SB(); \
    MMA(0, LPOS, 32); \
    MMA_RD(1, nx, 0); \
    p1 += H_BYTES; p1 = p1 == re1 ? rb1 : p1; \
    MOVB(1, p1 - p0); \
    MMA(1, 20, 32); \
  } while (0)

  BAR();
#pragma unroll
  for (int i = 0; i < 16; ++i) RD(0, 0, 0, i);
  SB();
#pragma unroll
  for (int i = 0; i < 4; ++i) offB[i] += p1 - p0;
  SB();
  for (int t = 0; t < NT - 1; t += 3) {
    TILE(0, 1);
    TILE(1, 2);
    TILE(2, 0);
  }
  MMA_RD(0, 0, 1); MMA(0, 16, 32);
  float mx0 = 0.f, mx1 = 0.f;
#define AMX(i) do { \
    asm volatile("v_max3_f32 %0, |%1|, |%2|, %0" : "+v"(mx0) : "v"(acc[MI(i)][NI(i)][0]), "v"(acc[MI(i)][NI(i)][1])); \
    asm volatile("v_max3_f32 %0, |%1|, |%2|, %0" : "+v"(mx1) : "v"(acc[MI(i)][NI(i)][2]), "v"(acc[MI(i)][NI(i)][3])); } while (0)
  MMA(1, 0, 8);
#pragma unroll
  for (int i = 8; i < 32; ++i) { MF(1, i); AMX(i - 8); SB(); }
#pragma unroll
  for (int i = 24; i < 32; ++i) { AMX(i); SB(); }
  if (wid == 0) STAMP(7);
  const unsigned mw = wave_umax(max(__float_as_uint(mx0), __float_as_uint(mx1)));
  if (lane == 0) ctl[C_RED + wid] = mw;
  char* orow = p.out[dst] + (size_t)(p.rank * MLOC + m0 + wm * 128 + wn * 8 + (lane >> 3)) * NSH + ct * BN + (lane & 7) * 16;
  LGKM0(); BAR();
  {
    const char* gp = myws + WS_AMAX + ((size_t)(call & 1) * NR + 2 * wid) * NWG * 8 + lane * 16;
    const unsigned long long t0 = __builtin_amdgcn_s_memrealtime();
    unsigned m = 0; bool gave_up = false;
    __builtin_amdgcn_s_sleep(PSLEEP0);
    for (;;) {
      u32x4 a, c, d, e; unsigned tw;
      asm volatile("global_load_dwordx4 %0, %5, off sc0 sc1\n\t"
                   "global_load_dwordx4 %1, %5, off offset:1024 sc0 sc1\n\t"
                   "global_load_dwordx4 %2, %5, off offset:2048 sc0 sc1\n\t"
                   "global_load_dwordx4 %3, %5, off offset:3072 sc0 sc1\n\t"
                   "global_load_dword %4, %6, off sc1\n\t"
                   "s_waitcnt vmcnt(0)"
                   : "=&v"(a), "=&v"(c), "=&v"(d), "=&v"(e), "=&v"(tw) : "v"(gp), "v"(tmo) : "memory");
      const bool ok = a.y == call && a.w == call && c.y == call && c.w == call && d.y == call && d.w == call && e.y == call && e.w == call;
      if (RFL(tw) == call) { gave_up = true; break; }
      if (__all(ok)) { m = max(max(max(a.x, a.z), max(c.x, c.z)), max(max(d.x, d.z), max(e.x, e.z))); break; }
      if (__builtin_amdgcn_s_memrealtime() - t0 > DEADLINE) { __hip_atomic_store(tmo, call, __ATOMIC_RELAXED, __HIP_MEMORY_SCOPE_AGENT); gave_up = true; break; }
      __builtin_amdgcn_s_sleep(PSLEEP);
    }
    m = wave_umax(m);
    if (lane == 0) { ctl[C_RED + 4 + wid] = m; ctl[C_BAD + wid] = gave_up ? 1u : 0u; }
  }
  LGKM0(); BAR();
  const float amax = __uint_as_float(max(max(ctl[C_RED + 4], ctl[C_RED + 5]), max(ctl[C_RED + 6], ctl[C_RED + 7])));
  const bool bad = (ctl[C_BAD] | ctl[C_BAD + 1] | ctl[C_BAD + 2] | ctl[C_BAD + 3]) != 0u;
  const float inv = 127.0f / amax;
  const float MAGIC = 12582912.0f;
  auto quant = [&](int m) {
    unsigned dw[4];
#pragma unroll
    for (int n = 0; n < 4; ++n) {
      unsigned q[4];
#pragma unroll
      for (int e = 0; e < 4; ++e)
        q[e] = __float_as_uint(__builtin_amdgcn_fmed3f(__builtin_fmaf(acc[m][n][e], inv, MAGIC), MAGIC - 127.0f, MAGIC + 127.0f));
      const unsigned lo = __builtin_amdgcn_perm(q[1], q[0], 0x0c0c0400u), hi = __builtin_amdgcn_perm(q[3], q[2], 0x04000c0cu);
      dw[n] = lo | hi;
    }
    auto s02 = __builtin_amdgcn_permlane32_swap(dw[0], dw[2], false, false);
    auto s13 = __builtin_amdgcn_permlane32_swap(dw[1], dw[3], false, false);
    auto s01 = __builtin_amdgcn_permlane16_swap(s02[0], s13[0], false, false);
    auto s23 = __builtin_amdgcn_permlane16_swap(s02[1], s13[1], false, false);
    u32x4 v = {s01[0], s01[1], s23[0], s23[1]};
    if (bad) v = (u32x4){0x80808080u, 0x80808080u, 0x80808080u, 0x80808080u};
    return v;
  };
  LAS char* xw = lds + LDS_XCH + (wm * 128 + fr) * 128 + (((wn * 4 + fq) ^ (fr & 7)) << 4);
  const LAS char* xr = lds + LDS_XCH + (wm * 128 + wn * 8 + (lane >> 3)) * 128 + (((lane & 7) ^ (lane >> 3)) << 4);
  *(LAS u32x4*)xw = quant(0);
  LGKM0(); BAR();
  store16_sys(orow, *(const LAS u32x4*)xr);
  SB();
#pragma unroll
  for (int m = 1; m < 8; ++m) *(LAS u32x4*)(xw + m * 2048) = quant(m);
  LGKM0(); BAR();
  u32x4 ov[8];
#pragma unroll
  for (int m = 1; m < 8; ++m) ov[m] = *(const LAS u32x4*)(xr + m * 2048);
#pragma unroll
  for (int m = 1; m < 8; ++m) {
    if (dst != p.rank) WAIT_V(0);
    store16_sys(orow + (size_t)m * 16 * NSH, ov[m]);
  }
  WAIT_V(0);
  BAR();
}
}

extern "C" void kernel_launch(int rank, int nranks, ncclComm_t comm, hipStream_t stream,
                              void* const* d_in, const int* in_sizes, int n_in,
                              void* d_out, int out_size, const int* in_dtypes, int out_dtype,
                              void* const* d_out_all, void* const* d_ws_all, size_t ws_bytes) {
  if ((nranks != NR && !SOLO_) || (unsigned)rank >= (unsigned)NR || ws_bytes < WS_NEED) { fprintf(stderr, "unsupported configuration\n"); return; }
  Params p;
  p.x = (const unsigned short*)d_in[0]; p.w = (const unsigned short*)d_in[1];
  for (int i = 0; i < NR; ++i) { p.out[i] = (char*)d_out_all[i]; p.ws[i] = (char*)d_ws_all[i]; }
  p.rank = rank; p.pad = 0;
  hipLaunchKernelGGL(gemm_a2a_q8, dim3(NWG), dim3(320), 0, stream, p);
  HIP_CHECK(hipGetLastError());
}
```

```cpp
#include <hip/hip_runtime.h>
#include <rccl/rccl.h>
#include <cstdio>
#include <cstdint>

#define HIP_CHECK(expr) do { hipError_t _e = (expr); if (_e != hipSuccess) { \
    fprintf(stderr, "HIP error %s at %s:%d\n", hipGetErrorString(_e), __FILE__, __LINE__); \
    return; } } while (0)

namespace {
#define LAS __attribute__((address_space(3)))
typedef short bf16x8 __attribute__((ext_vector_type(8)));
typedef short s16x4 __attribute__((ext_vector_type(4)));
typedef float f32x4 __attribute__((ext_vector_type(4)));
typedef unsigned u32x4 __attribute__((ext_vector_type(4)));

constexpr int NR = 8;
constexpr int MLOC = 512, KD = 4096, ND = 16384, NSH = ND / NR;
constexpr int BM = 256, BN = 128, BK = 64, NT = KD / BK;
constexpr int NWG = 256;
constexpr int A_BYTES = BM * BK * 2, H_BYTES = BK * BN, NAS = 3, NLS = 5, NFS = 3;
constexpr int LDS_A = 0, LDS_L = NAS * A_BYTES, LDS_F = LDS_L + NLS * H_BYTES, LDS_BYTES = LDS_F + NFS * H_BYTES;
constexpr int LDS_CTL = LDS_A + A_BYTES;
constexpr int LDS_XCH = LDS_A + 2 * A_BYTES;
constexpr int C_RED = 0;
constexpr int C_BAD = 8;
constexpr size_t WS_CNT = 0;
constexpr size_t WS_TMO = 4096;
constexpr size_t WS_AMAX = 8192;
constexpr size_t WS_DONE = 65536;
constexpr size_t WS_NEED = 1u << 20;
constexpr unsigned long long DEADLINE = 50000000ull;

struct Params {
  const unsigned short* x; const unsigned short* w;
  char* out[NR]; char* ws[NR];
  int rank; int pad;
};

#define WAIT_V(n) asm volatile("s_waitcnt vmcnt(%0)" :: "i"(n) : "memory")
#define BAR() do { asm volatile("" ::: "memory"); __builtin_amdgcn_s_barrier(); asm volatile("" ::: "memory"); } while (0)
#define LGKM0() __builtin_amdgcn_s_waitcnt(0xC07F)
#define RFL(x) __builtin_amdgcn_readfirstlane(x)
#define SB() __builtin_amdgcn_sched_barrier(0)

__device__ __forceinline__ void store16_sys(void* ptr, u32x4 v) {
  asm volatile("global_store_dwordx4 %0, %1, off sc0 sc1\n\ts_nop 1" :: "v"(ptr), "v"(v) : "memory");
}
__device__ __forceinline__ unsigned wave_umax(unsigned v) {
  v = max(v, (unsigned)__builtin_amdgcn_update_dpp(0, (int)v, 0xB1, 0xF, 0xF, true));
  v = max(v, (unsigned)__builtin_amdgcn_update_dpp(0, (int)v, 0x4E, 0xF, 0xF, true));
  v = max(v, (unsigned)__builtin_amdgcn_update_dpp(0, (int)v, 0x141, 0xF, 0xF, true));
  v = max(v, (unsigned)__builtin_amdgcn_update_dpp(0, (int)v, 0x140, 0xF, 0xF, true));
  auto a = __builtin_amdgcn_permlane16_swap(v, v, false, false); v = max(a[0], a[1]);
  auto c = __builtin_amdgcn_permlane32_swap(v, v, false, false); return max(c[0], c[1]);
}
#define GL8(base, v, dst) \
    asm volatile("s_mov_b32 m0, %1\n\ts_nop 0\n\t" \
                 "global_load_lds_dwordx4 %2, %0 offset:-4096\n\tglobal_load_lds_dwordx4 %3, %0 offset:-3072\n\t" \
                 "global_load_lds_dwordx4 %4, %0 offset:-2048\n\tglobal_load_lds_dwordx4 %5, %0 offset:-1024\n\t" \
                 "global_load_lds_dwordx4 %6, %0\n\tglobal_load_lds_dwordx4 %7, %0 offset:1024\n\t" \
                 "global_load_lds_dwordx4 %8, %0 offset:2048\n\tglobal_load_lds_dwordx4 %9, %0 offset:3072" \
                 :: "s"(base), "s"((dst) + 4096), "v"((v)[0]), "v"((v)[1]), "v"((v)[2]), "v"((v)[3]), "v"((v)[4]), "v"((v)[5]), "v"((v)[6]), "v"((v)[7]) : "memory")

__global__ __launch_bounds__(320, 2) void gemm_a2a_q8(const Params p) {
  __shared__ __attribute__((aligned(16))) char smem[LDS_BYTES];
  LAS char* lds = (LAS char*)smem;
  LAS unsigned* ctl = (LAS unsigned*)(lds + LDS_CTL);
  const int tid = threadIdx.x, lane = tid & 63;
  const int wid = RFL(tid >> 6);
  const int b = blockIdx.x, xg = b & 7, j = b >> 3;
  const int dst = (xg + p.rank) & 7;
  const int ct = j >> 1, rt = j & 1, m0 = rt * BM;
  const int n0 = dst * NSH + ct * BN;
  const int kt0 = xg * 8;
  char* const myws = p.ws[p.rank];
  unsigned* const cnt = (unsigned*)(myws + WS_CNT) + b;
  unsigned* const tmo = (unsigned*)(myws + WS_TMO);
#define LOAD_CALL(call) unsigned call; asm volatile("s_load_dword %0, %1, 0x0\n\ts_waitcnt lgkmcnt(0)" : "=s"(call) : "s"(cnt) : "memory"); call += 1

  if (wid == 4) {
    const unsigned ldsb = (unsigned)(uintptr_t)lds;
    unsigned va[8], vb[8];
#pragma unroll
    for (int i = 0; i < 8; ++i) {
      const int kr = 8 * (i >> 1) + (lane >> 3);
      vb[i] = (unsigned)(kr * (ND * 2) + (i & 1) * 128 + (((lane & 7) ^ ((((i >> 1) & 1) << 2) | ((lane >> 3) & 2))) << 4)) + 8192u - i * 1024u;
      const int row = 8 * i + (lane >> 3);
      va[i] = (unsigned)(row * (KD * 2) + (((lane & 7) ^ ((row >> 1) & 7)) << 4)) + 8192u - i * 1024u;
    }
    const char* abase = (const char*)p.x + (size_t)m0 * (KD * 2) - 4096;
    const char* lbase = (const char*)p.w + (size_t)n0 * 2 + (size_t)rt * (32 * ND * 2) - 4096;
    const char* fbase = (const char*)p.w + (size_t)n0 * 2 + (size_t)(1 - rt) * (32 * ND * 2) - 4096;
    auto issue_ah = [&](int t, int slot, int h) {
      const char* at = abase + ((t + kt0) & (NT - 1)) * (BK * 2) + (size_t)h * (128 * KD * 2);
#pragma unroll
      for (int g = 0; g < 2; ++g) GL8(at + (size_t)g * (64 * KD * 2), va, ldsb + LDS_A + slot * A_BYTES + h * 16384 + g * 8192);
    };
    auto issue_a = [&](int t, int slot) { issue_ah(t, slot, 0); issue_ah(t, slot, 1); };
    auto issue_l = [&](int t, int slot) { GL8(lbase + (unsigned)(((t + kt0) & (NT - 1)) * (BK * ND * 2)), vb, ldsb + LDS_L + slot * H_BYTES); };
    auto issue_f = [&](int t, int slot) { GL8(fbase + (unsigned)(((t + kt0) & (NT - 1)) * (BK * ND * 2)), vb, ldsb + LDS_F + slot * H_BYTES); };
    for (int i = 0; i < (ct & 7); ++i) __builtin_amdgcn_s_sleep(2);
    issue_l(0, 0); issue_f(0, 0); issue_a(0, 0); issue_l(1, 1); issue_f(1, 1);
    LOAD_CALL(call);
    char* pw = p.ws[0];
#pragma unroll
    for (int i = 1; i < NR; ++i) pw = (lane & 7) == i ? p.ws[i] : pw;
    unsigned long long* gran = (unsigned long long*)(pw + WS_AMAX) + ((size_t)(call & 1) * NR + p.rank) * NWG + b;
    asm volatile("" : "+v"(gran));
    WAIT_V(16); BAR();
    issue_a(1, 1); issue_l(2, 2); issue_f(2, 2); issue_ah(2, 2, 0);
    WAIT_V(32); BAR();
    issue_ah(2, 2, 1); issue_l(3, 3);
    int sa = 0, sl = 4, tl = 4, prev_l = 8;
    for (int t = 1; t < NT - 1; ++t) {
      int n = 0, nl = 0;
      if (t + 2 < NT) { issue_a(t + 2, sa); issue_f(t + 2, sa); sa = sa == NAS - 1 ? 0 : sa + 1; n = 40; }
      if (t & 1) {
#pragma unroll
        for (int i = 0; i < 2; ++i)
          if (tl < NT) { issue_l(tl, sl); ++tl; sl = sl == NLS - 1 ? 0 : sl + 1; nl += 8; }
      }
      n += nl + prev_l; prev_l = nl;
      if (n >= 56) WAIT_V(56); else if (n >= 40) WAIT_V(40); else WAIT_V(0);
      BAR();
    }
    BAR();
    const unsigned m = max(max(ctl[C_RED], ctl[C_RED + 1]), max(ctl[C_RED + 2], ctl[C_RED + 3]));
    if (lane < NR) __hip_atomic_store(gran, ((unsigned long long)call << 32) | m, __ATOMIC_RELAXED, __HIP_MEMORY_SCOPE_SYSTEM);
    if (lane == 0) __hip_atomic_store(cnt, call, __ATOMIC_RELAXED, __HIP_MEMORY_SCOPE_AGENT);
    BAR();
    BAR(); BAR();
    BAR();
    if (lane == 0 && dst != p.rank)
      __hip_atomic_store((unsigned*)(p.ws[dst] + WS_DONE) + p.rank * 32 + j, call, __ATOMIC_RELAXED, __HIP_MEMORY_SCOPE_SYSTEM);
    if (b == 0) {
      const char* fp = myws + WS_DONE + lane * 16;
      const bool mine = (lane >> 3) == p.rank;
      const unsigned long long t0 = __builtin_amdgcn_s_memrealtime();
      for (;;) {
        u32x4 a; unsigned tw;
        asm volatile("global_load_dwordx4 %0, %2, off sc0 sc1\n\tglobal_load_dword %1, %3, off sc1\n\ts_waitcnt vmcnt(0)" : "=&v"(a), "=&v"(tw) : "v"(fp), "v"(tmo) : "memory");
        const bool ok = mine || (a.x >= call && a.y >= call && a.z >= call && a.w >= call);
        if (__all(ok)) break;
        if (RFL(tw) == call || __builtin_amdgcn_s_memrealtime() - t0 > DEADLINE) {
          __hip_atomic_store(tmo, call, __ATOMIC_RELAXED, __HIP_MEMORY_SCOPE_AGENT);
          if (lane == 0) p.out[p.rank][(size_t)(p.rank * MLOC) * NSH] = (char)-128;
          break;
        }
        __builtin_amdgcn_s_sleep(1);
      }
    }
    return;
  }

  const int wm = wid >> 1, wn = wid & 1, fr = lane & 15, fq = lane >> 4;
  LOAD_CALL(call);
  f32x4 acc[8][4];
#pragma unroll
  for (int m = 0; m < 8; ++m)
#pragma unroll
    for (int n = 0; n < 4; ++n) acc[m][n] = (f32x4){0.f, 0.f, 0.f, 0.f};
  const int rb0 = rt == 0 ? LDS_L : LDS_F, rb1 = rt == 0 ? LDS_F : LDS_L;
  const int re0 = rb0 + (rt == 0 ? NLS : NFS) * H_BYTES, re1 = rb1 + (rt == 0 ? NFS : NLS) * H_BYTES;
  int offA[2], offAh[2], offB[4];
#pragma unroll
  for (int ks = 0; ks < 2; ++ks) { offA[ks] = LDS_A + (wm * 128 + fr) * 128 + (((4 * ks + fq) ^ ((fr >> 1) & 7)) << 4); offAh[ks] = offA[ks] + 2 * A_BYTES; }
  {
    const int q = (lane & 15) >> 2, pp = lane & 3;
#pragma unroll
    for (int n = 0; n < 4; ++n)
      offB[n] = rb0 + 2048 * fq + 1024 * wn + 128 * q + (((2 * n + (pp >> 1)) ^ (((fq & 1) << 2) | (q & 2))) << 4) + 8 * (pp & 1);
  }
  asm volatile("" : "+v"(offAh[0]), "+v"(offAh[1]));
  bf16x8 Af[2][8];
  s16x4 Bl[2][4], Bh[2][4];
#define RD(buf, sl, ks, i) do { \
    if ((i) < 8) { \
      const s16x4 v_ = __builtin_amdgcn_ds_read_tr16_b64_v4i16((LAS s16x4*)(lds + offB[((i) >> 1) & 3] + ((i) & 1) * 512)); \
      if ((i) & 1) Bh[buf][((i) >> 1) & 3] = v_; else Bl[buf][((i) >> 1) & 3] = v_; \
    } else Af[buf][(i) & 7] = *(const LAS bf16x8*)(lds + ((sl) == 2 ? offAh[ks] : offA[ks] + (sl) * A_BYTES) + ((i) & 7) * 2048); \
  } while (0)
#define MI(i) ((i) >> 2)
#define NI(i) ((((i) >> 2) & 1) ? 3 - ((i) & 3) : ((i) & 3))
#define MF(buf, i) asm volatile("v_mfma_f32_16x16x32_bf16 %0, %1, %2, %0" : "+v"(acc[MI(i)][NI(i)]) \
    : "v"(__builtin_shufflevector(Bl[buf][NI(i)], Bh[buf][NI(i)], 0, 1, 2, 3, 4, 5, 6, 7)), "v"(Af[buf][MI(i)]))
#define MMA(buf, i0, i1) do { _Pragma("unroll") for (int i = (i0); i < (i1); ++i) { MF(buf, i); SB(); } } while (0)
#define MMA_RD(buf, sl, ks) do { _Pragma("unroll") for (int i = 0; i < 16; ++i) { MF(buf, i); RD(1 - (buf), sl, ks, i); SB(); } } while (0)
#define MOVB(buf) do { _Pragma("unroll") for (int i = 0; i < 4; ++i) { MF(buf, 16 + i); offB[i] += dd; asm volatile("" : "+v"(offB[i])); SB(); } } while (0)
#define NEXTP(buf, pp, rb, re, po) do { \
    MF(buf, 20); pp += H_BYTES; asm volatile("" : "+s"(pp)); SB(); \
    MF(buf, 21); pp = pp == (re) ? (rb) : pp; asm volatile("" : "+s"(pp)); SB(); \
    MF(buf, 22); dd = pp - (po); asm volatile("" : "+s"(dd)); SB(); } while (0)
#define TILE(sl, nx) do { \
    MMA_RD(0, sl, 1); \
    MOVB(0); \
    NEXTP(0, p1, rb1, re1, p0); \
    MMA(0, 23, 31); \
    LGKM0(); BAR(); SB();              \
    MMA(0, 31, 32); \
    MMA_RD(1, nx, 0); \
    MOVB(1); \
    NEXTP(1, p0, rb0, re0, p1); \
    MMA(1, 23, 32); \
  } while (0)

#pragma unroll
  for (int m = 0; m < 8; ++m) asm volatile("" : "+v"(acc[m][0]), "+v"(acc[m][1]), "+v"(acc[m][2]), "+v"(acc[m][3]));
  BAR();
#pragma unroll
  for (int i = 0; i < 16; ++i) RD(0, 0, 0, i);
  SB();
#pragma unroll
  for (int i = 0; i < 4; ++i) offB[i] += rb1 - rb0;
  int p0 = rb0 + H_BYTES, p1 = rb1, dd = p0 - p1;
  SB();
  for (int t = 0; t < NT - 1; t += 3) {
    TILE(0, 1);
    TILE(1, 2);
    TILE(2, 0);
  }
  MMA_RD(0, 0, 1); MMA(0, 16, 32);
  float mx0 = 0.f, mx1 = 0.f;
#define AMX(i) do { \
    asm volatile("v_max3_f32 %0, |%1|, |%2|, %0" : "+v"(mx0) : "v"(acc[MI(i)][NI(i)][0]), "v"(acc[MI(i)][NI(i)][1])); \
    asm volatile("v_max3_f32 %0, |%1|, |%2|, %0" : "+v"(mx1) : "v"(acc[MI(i)][NI(i)][2]), "v"(acc[MI(i)][NI(i)][3])); } while (0)
  MMA(1, 0, 8);
#pragma unroll
  for (int i = 8; i < 32; ++i) { MF(1, i); AMX(i - 8); SB(); }
#pragma unroll
  for (int i = 24; i < 32; ++i) { AMX(i); SB(); }
  const unsigned mw = wave_umax(max(__float_as_uint(mx0), __float_as_uint(mx1)));
  if (lane == 0) ctl[C_RED + wid] = mw;
  char* orow = p.out[dst] + (size_t)(p.rank * MLOC + m0 + wm * 128 + wn * 8 + (lane >> 3)) * NSH + ct * BN + (lane & 7) * 16;
  LGKM0(); BAR();
  {
    const char* gp = myws + WS_AMAX + ((size_t)(call & 1) * NR + 2 * wid) * NWG * 8 + lane * 16;
    const unsigned long long t0 = __builtin_amdgcn_s_memrealtime();
    unsigned m = 0; bool gave_up = false;
    __builtin_amdgcn_s_sleep(12);
    for (;;) {
      u32x4 a, c, d, e; unsigned tw;
      asm volatile("global_load_dwordx4 %0, %5, off sc0 sc1\n\t"
                   "global_load_dwordx4 %1, %5, off offset:1024 sc0 sc1\n\t"
                   "global_load_dwordx4 %2, %5, off offset:2048 sc0 sc1\n\t"
                   "global_load_dwordx4 %3, %5, off offset:3072 sc0 sc1\n\t"
                   "global_load_dword %4, %6, off sc1\n\t"
                   "s_waitcnt vmcnt(0)"
                   : "=&v"(a), "=&v"(c), "=&v"(d), "=&v"(e), "=&v"(tw) : "v"(gp), "v"(tmo) : "memory");
      const bool ok = a.y == call && a.w == call && c.y == call && c.w == call && d.y == call && d.w == call && e.y == call && e.w == call;
      if (RFL(tw) == call) { gave_up = true; break; }
      if (__all(ok)) { m = max(max(max(a.x, a.z), max(c.x, c.z)), max(max(d.x, d.z), max(e.x, e.z))); break; }
      if (__builtin_amdgcn_s_memrealtime() - t0 > DEADLINE) { __hip_atomic_store(tmo, call, __ATOMIC_RELAXED, __HIP_MEMORY_SCOPE_AGENT); gave_up = true; break; }
      __builtin_amdgcn_s_sleep(6);
    }
    m = wave_umax(m);
    if (lane == 0) { ctl[C_RED + 4 + wid] = m; ctl[C_BAD + wid] = gave_up ? 1u : 0u; }
  }
  LGKM0(); BAR();
  const float amax = __uint_as_float(max(max(ctl[C_RED + 4], ctl[C_RED + 5]), max(ctl[C_RED + 6], ctl[C_RED + 7])));
  const bool bad = (ctl[C_BAD] | ctl[C_BAD + 1] | ctl[C_BAD + 2] | ctl[C_BAD + 3]) != 0u;
  const float inv = 127.0f / amax;
  const float MAGIC = 12582912.0f;
  auto quant = [&](int m) {
    unsigned dw[4];
#pragma unroll
    for (int n = 0; n < 4; ++n) {
      unsigned q[4];
#pragma unroll
      for (int e = 0; e < 4; ++e)
        q[e] = __float_as_uint(__builtin_amdgcn_fmed3f(__builtin_fmaf(acc[m][n][e], inv, MAGIC), MAGIC - 127.0f, MAGIC + 127.0f));
      const unsigned lo = __builtin_amdgcn_perm(q[1], q[0], 0x0c0c0400u), hi = __builtin_amdgcn_perm(q[3], q[2], 0x04000c0cu);
      dw[n] = lo | hi;
    }
    auto s02 = __builtin_amdgcn_permlane32_swap(dw[0], dw[2], false, false);
    auto s13 = __builtin_amdgcn_permlane32_swap(dw[1], dw[3], false, false);
    auto s01 = __builtin_amdgcn_permlane16_swap(s02[0], s13[0], false, false);
    auto s23 = __builtin_amdgcn_permlane16_swap(s02[1], s13[1], false, false);
    u32x4 v = {s01[0], s01[1], s23[0], s23[1]};
    if (bad) v = (u32x4){0x80808080u, 0x80808080u, 0x80808080u, 0x80808080u};
    return v;
  };
  LAS char* xw = lds + LDS_XCH + (wm * 128 + fr) * 128 + (((wn * 4 + fq) ^ (fr & 7)) << 4);
  const LAS char* xr = lds + LDS_XCH + (wm * 128 + wn * 8 + (lane >> 3)) * 128 + (((lane & 7) ^ (lane >> 3)) << 4);
  *(LAS u32x4*)xw = quant(0);
  LGKM0(); BAR();
  store16_sys(orow, *(const LAS u32x4*)xr);
  SB();
#pragma unroll
  for (int m = 1; m < 8; ++m) *(LAS u32x4*)(xw + m * 2048) = quant(m);
  LGKM0(); BAR();
  u32x4 ov[8];
#pragma unroll
  for (int m = 1; m < 8; ++m) ov[m] = *(const LAS u32x4*)(xr + m * 2048);
#pragma unroll
  for (int m = 1; m < 8; ++m) {
    if (dst != p.rank) WAIT_V(0);
    store16_sys(orow + (size_t)m * 16 * NSH, ov[m]);
  }
  WAIT_V(0);
  BAR();
}
}

extern "C" void kernel_launch(int rank, int nranks, ncclComm_t comm, hipStream_t stream,
                              void* const* d_in, const int* in_sizes, int n_in,
                              void* d_out, int out_size, const int* in_dtypes, int out_dtype,
                              void* const* d_out_all, void* const* d_ws_all, size_t ws_bytes) {
  if (nranks != NR || (unsigned)rank >= (unsigned)NR || ws_bytes < WS_NEED || n_in != 2 || in_sizes[0] != MLOC * KD || in_sizes[1] != KD * ND ||
      out_size != NR * MLOC * NSH) { fprintf(stderr, "unsupported configuration\n"); return; }
  Params p;
  p.x = (const unsigned short*)d_in[0]; p.w = (const unsigned short*)d_in[1];
  for (int i = 0; i < NR; ++i) { p.out[i] = (char*)d_out_all[i]; p.ws[i] = (char*)d_ws_all[i]; }
  p.rank = rank; p.pad = 0;
  hipLaunchKernelGGL(gemm_a2a_q8, dim3(NWG), dim3(320), 0, stream, p);
  HIP_CHECK(hipGetLastError());
}
```

```cpp
#include <hip/hip_runtime.h>
#include <rccl/rccl.h>
#include <cstdio>
#include <cstdint>

#define HIP_CHECK(expr) do { hipError_t _e = (expr); if (_e != hipSuccess) { \
    fprintf(stderr, "HIP error %s at %s:%d\n", hipGetErrorString(_e), __FILE__, __LINE__); \
    return; } } while (0)

namespace {
#define LAS __attribute__((address_space(3)))
typedef short bf16x8 __attribute__((ext_vector_type(8)));
typedef short s16x4 __attribute__((ext_vector_type(4)));
typedef float f32x4 __attribute__((ext_vector_type(4)));
typedef unsigned u32x4 __attribute__((ext_vector_type(4)));

constexpr int NR = 8;
constexpr int MLOC = 512, KD = 4096, ND = 16384, NSH = ND / NR;
constexpr int BM = 256, BN = 128, BK = 64, NT = KD / BK;
constexpr int NWG = 256;
constexpr int A_BYTES = BM * BK * 2, H_BYTES = BK * BN, NAS = 3, NLS = 5, NFS = 3;
constexpr int LDS_A = 0, LDS_L = NAS * A_BYTES, LDS_F = LDS_L + NLS * H_BYTES, LDS_BYTES = LDS_F + NFS * H_BYTES;
constexpr int LDS_CTL = LDS_A + A_BYTES;
constexpr int LDS_XCH = LDS_A + 2 * A_BYTES;
constexpr int C_RED = 0;
constexpr int C_BAD = 8;
constexpr size_t WS_CNT = 0;
constexpr size_t WS_TMO = 4096;
constexpr size_t WS_AMAX = 8192;
constexpr size_t WS_DONE = 65536;
constexpr size_t WS_NEED = 1u << 20;
constexpr unsigned long long DEADLINE = 50000000ull;

struct Params {
  const unsigned short* x; const unsigned short* w;
  char* out[NR]; char* ws[NR];
  int rank; int pad;
};

#define WAIT_V(n) asm volatile("s_waitcnt vmcnt(%0)" :: "i"(n) : "memory")
#define BAR() do { asm volatile("" ::: "memory"); __builtin_amdgcn_s_barrier(); asm volatile("" ::: "memory"); } while (0)
#define LGKM0() __builtin_amdgcn_s_waitcnt(0xC07F)
#define RFL(x) __builtin_amdgcn_readfirstlane(x)
#define SB() __builtin_amdgcn_sched_barrier(0)

__device__ __forceinline__ void store16_sys(void* ptr, u32x4 v) {
  asm volatile("global_store_dwordx4 %0, %1, off sc0 sc1\n\ts_nop 1" :: "v"(ptr), "v"(v) : "memory");
}
__device__ __forceinline__ unsigned wave_umax(unsigned v) {
  v = max(v, (unsigned)__builtin_amdgcn_update_dpp(0, (int)v, 0xB1, 0xF, 0xF, true));
  v = max(v, (unsigned)__builtin_amdgcn_update_dpp(0, (int)v, 0x4E, 0xF, 0xF, true));
  v = max(v, (unsigned)__builtin_amdgcn_update_dpp(0, (int)v, 0x141, 0xF, 0xF, true));
  v = max(v, (unsigned)__builtin_amdgcn_update_dpp(0, (int)v, 0x140, 0xF, 0xF, true));
  auto a = __builtin_amdgcn_permlane16_swap(v, v, false, false); v = max(a[0], a[1]);
  auto c = __builtin_amdgcn_permlane32_swap(v, v, false, false); return max(c[0], c[1]);
}
#define GL8(base, v, dst) \
    asm volatile("s_mov_b32 m0, %1\n\ts_nop 0\n\t" \
                 "global_load_lds_dwordx4 %2, %0 offset:-4096\n\tglobal_load_lds_dwordx4 %3, %0 offset:-3072\n\t" \
                 "global_load_lds_dwordx4 %4, %0 offset:-2048\n\tglobal_load_lds_dwordx4 %5, %0 offset:-1024\n\t" \
                 "global_load_lds_dwordx4 %6, %0\n\tglobal_load_lds_dwordx4 %7, %0 offset:1024\n\t" \
                 "global_load_lds_dwordx4 %8, %0 offset:2048\n\tglobal_load_lds_dwordx4 %9, %0 offset:3072" \
                 :: "s"(base), "s"((dst) + 4096), "v"((v)[0]), "v"((v)[1]), "v"((v)[2]), "v"((v)[3]), "v"((v)[4]), "v"((v)[5]), "v"((v)[6]), "v"((v)[7]) : "memory")

__global__ __launch_bounds__(320, 2) void gemm_a2a_q8(const Params p) {
  __shared__ __attribute__((aligned(16))) char smem[LDS_BYTES];
  LAS char* lds = (LAS char*)smem;
  LAS unsigned* ctl = (LAS unsigned*)(lds + LDS_CTL);
  const int tid = threadIdx.x, lane = tid & 63;
  const int wid = RFL(tid >> 6);
  const int b = blockIdx.x, xg = b & 7, j = b >> 3;
  const int dst = (xg + p.rank) & 7;
  const int ct = j >> 1, rt = j & 1, m0 = rt * BM;
  const int n0 = dst * NSH + ct * BN;
  const int kt0 = xg * 8;
  char* const myws = p.ws[p.rank];
  unsigned* const cnt = (unsigned*)(myws + WS_CNT) + b;
  unsigned* const tmo = (unsigned*)(myws + WS_TMO);
#define LOAD_CALL(call) unsigned call; asm volatile("s_load_dword %0, %1, 0x0\n\ts_waitcnt lgkmcnt(0)" : "=s"(call) : "s"(cnt) : "memory"); call += 1

  if (wid == 4) {
    const unsigned ldsb = (unsigned)(uintptr_t)lds;
    unsigned va[8], vb[8];
#pragma unroll
    for (int i = 0; i < 8; ++i) {
      const int kr = 8 * (i >> 1) + (lane >> 3);
      vb[i] = (unsigned)(kr * (ND * 2) + (i & 1) * 128 + (((lane & 7) ^ ((((i >> 1) & 1) << 2) | ((lane >> 3) & 2))) << 4)) + 8192u - i * 1024u;
      const int row = 8 * i + (lane >> 3);
      va[i] = (unsigned)(row * (KD * 2) + (((lane & 7) ^ ((row >> 1) & 7)) << 4)) + 8192u - i * 1024u;
    }
    const char* abase = (const char*)p.x + (size_t)m0 * (KD * 2) - 4096;
    const char* lbase = (const char*)p.w + (size_t)n0 * 2 + (size_t)rt * (32 * ND * 2) - 4096;
    const char* fbase = (const char*)p.w + (size_t)n0 * 2 + (size_t)(1 - rt) * (32 * ND * 2) - 4096;
    auto issue_ah = [&](int t, int slot, int h) {
      const char* at = abase + ((t + kt0) & (NT - 1)) * (BK * 2) + (size_t)h * (128 * KD * 2);
#pragma unroll
      for (int g = 0; g < 2; ++g) GL8(at + (size_t)g * (64 * KD * 2), va, ldsb + LDS_A + slot * A_BYTES + h * 16384 + g * 8192);
    };
    auto issue_a = [&](int t, int slot) { issue_ah(t, slot, 0); issue_ah(t, slot, 1); };
    auto issue_l = [&](int t, int slot) { GL8(lbase + (unsigned)(((t + kt0) & (NT - 1)) * (BK * ND * 2)), vb, ldsb + LDS_L + slot * H_BYTES); };
    auto issue_f = [&](int t, int slot) { GL8(fbase + (unsigned)(((t + kt0) & (NT - 1)) * (BK * ND * 2)), vb, ldsb + LDS_F + slot * H_BYTES); };
    for (int i = 0; i < (ct & 7); ++i) __builtin_amdgcn_s_sleep(2);
    issue_l(0, 0); issue_f(0, 0); issue_a(0, 0); issue_l(1, 1); issue_f(1, 1);
    LOAD_CALL(call);
    char* pw = p.ws[0];
#pragma unroll
    for (int i = 1; i < NR; ++i) pw = (lane & 7) == i ? p.ws[i] : pw;
    unsigned long long* gran = (unsigned long long*)(pw + WS_AMAX) + ((size_t)(call & 1) * NR + p.rank) * NWG + b;
    asm volatile("" : "+v"(gran));
    WAIT_V(0); BAR();
    issue_a(1, 1); issue_l(2, 2); issue_f(2, 2); issue_ah(2, 2, 0);
    WAIT_V(16); BAR();
    issue_ah(2, 2, 1); issue_l(3, 3); issue_f(3, 0); issue_l(4, 4);
    int sa = 0, sf = 1, sl = 0, tl = 5, prev_l = 8;
    for (int t = 1; t < NT - 1; ++t) {
      int n = 0, nl = 0;
      if (t + 2 < NT) { issue_a(t + 2, sa); sa = sa == NAS - 1 ? 0 : sa + 1; n = 32; }
      if (t + 3 < NT) { issue_f(t + 3, sf); sf = sf == NFS - 1 ? 0 : sf + 1; n += 8; }
      if (t & 1) {
#pragma unroll
        for (int i = 0; i < 2; ++i)
          if (tl < NT) { issue_l(tl, sl); ++tl; sl = sl == NLS - 1 ? 0 : sl + 1; nl += 8; }
      }
      n += nl + prev_l; prev_l = nl;
      if (n >= 56) WAIT_V(56); else if (n >= 40) WAIT_V(40); else if (n >= 32) WAIT_V(32); else WAIT_V(0);
      BAR();
    }
    BAR();
    const unsigned m = max(max(ctl[C_RED], ctl[C_RED + 1]), max(ctl[C_RED + 2], ctl[C_RED + 3]));
    if (lane < NR) __hip_atomic_store(gran, ((unsigned long long)call << 32) | m, __ATOMIC_RELAXED, __HIP_MEMORY_SCOPE_SYSTEM);
    if (lane == 0) __hip_atomic_store(cnt, call, __ATOMIC_RELAXED, __HIP_MEMORY_SCOPE_AGENT);
    BAR();
    BAR(); BAR();
    BAR();
    if (lane == 0 && dst != p.rank)
      __hip_atomic_store((unsigned*)(p.ws[dst] + WS_DONE) + p.rank * 32 + j, call, __ATOMIC_RELAXED, __HIP_MEMORY_SCOPE_SYSTEM);
    if (b == 0) {
      const char* fp = myws + WS_DONE + lane * 16;
      const bool mine = (lane >> 3) == p.rank;
      const unsigned long long t0 = __builtin_amdgcn_s_memrealtime();
      for (;;) {
        u32x4 a; unsigned tw;
        asm volatile("global_load_dwordx4 %0, %2, off sc0 sc1\n\tglobal_load_dword %1, %3, off sc1\n\ts_waitcnt vmcnt(0)" : "=&v"(a), "=&v"(tw) : "v"(fp), "v"(tmo) : "memory");
        const bool ok = mine || (a.x >= call && a.y >= call && a.z >= call && a.w >= call);
        if (__all(ok)) break;
        if (RFL(tw) == call || __builtin_amdgcn_s_memrealtime() - t0 > DEADLINE) {
          __hip_atomic_store(tmo, call, __ATOMIC_RELAXED, __HIP_MEMORY_SCOPE_AGENT);
          if (lane == 0) p.out[p.rank][(size_t)(p.rank * MLOC) * NSH] = (char)-128;
          break;
        }
        __builtin_amdgcn_s_sleep(1);
      }
    }
    return;
  }

  const int wm = wid >> 1, wn = wid & 1, fr = lane & 15, fq = lane >> 4;
  LOAD_CALL(call);
  f32x4 acc[8][4];
#pragma unroll
  for (int m = 0; m < 8; ++m)
#pragma unroll
    for (int n = 0; n < 4; ++n) acc[m][n] = (f32x4){0.f, 0.f, 0.f, 0.f};
  const int rb0 = rt == 0 ? LDS_L : LDS_F, rb1 = rt == 0 ? LDS_F : LDS_L;
  const int re0 = rb0 + (rt == 0 ? NLS : NFS) * H_BYTES, re1 = rb1 + (rt == 0 ? NFS : NLS) * H_BYTES;
  int offA[2], offB[4];
#pragma unroll
  for (int ks = 0; ks < 2; ++ks) offA[ks] = LDS_A + (wm * 128 + fr) * 128 + (((4 * ks + fq) ^ ((fr >> 1) & 7)) << 4);
  {
    const int q = (lane & 15) >> 2, pp = lane & 3;
#pragma unroll
    for (int n = 0; n < 4; ++n)
      offB[n] = rb0 + 2048 * fq + 1024 * wn + 128 * q + (((2 * n + (pp >> 1)) ^ (((fq & 1) << 2) | (q & 2))) << 4) + 8 * (pp & 1);
  }
  bf16x8 Af[4][2];
  s16x4 Bl[2][2][4], Bh[2][2][4];
#define RDA(m, ks) Af[(m) & 3][ks] = *(const LAS bf16x8*)(lds + offA[ks] + (m) * 2048)
#define RDB(pb, ks, i) do { \
    const s16x4 v_ = __builtin_amdgcn_ds_read_tr16_b64_v4i16((LAS s16x4*)(lds + offB[(i) >> 1] + ((i) & 1) * 512)); \
    if ((i) & 1) Bh[pb][ks][(i) >> 1] = v_; else Bl[pb][ks][(i) >> 1] = v_; } while (0)
#define MI(i) ((i) >> 3)
#define NI(i) ((((i) >> 3) & 1) ? 3 - (((i) >> 1) & 3) : (((i) >> 1) & 3))
#define KI(i) ((((i) >> 1) ^ (i)) & 1)
#define MF(pb, i) asm volatile("v_mfma_f32_16x16x32_bf16 %0, %1, %2, %0" : "+v"(acc[MI(i)][NI(i)]) \
    : "v"(__builtin_shufflevector(Bl[pb][KI(i)][NI(i)], Bh[pb][KI(i)][NI(i)], 0, 1, 2, 3, 4, 5, 6, 7)), "v"(Af[MI(i) & 3][KI(i)]))
#define PINV(x) asm volatile("" : "+v"(x))
#define PINS(x) asm volatile("" : "+s"(x))
#define GAP(pb, i) do { \
    constexpr int r_ = (i) >> 3, g_ = (i) & 7; \
    if (r_ < 5 && g_ < 2) RDA(r_ + 3, g_); \
    else if (r_ < 2 && g_ < 6) RDB(1 - (pb), 0, 4 * r_ + g_ - 2); \
    else if (r_ == 2 && g_ < 6) { offB[g_ - 2] += dd; PINV(offB[g_ - 2]); } \
    else if (r_ == 2 && g_ == 6) { p0 += H_BYTES; PINS(p0); } \
    else if (r_ == 2 && g_ == 7) { p0 = p0 == re0 ? rb0 : p0; PINS(p0); } \
    else if ((r_ == 3 || r_ == 4) && g_ < 6) RDB(1 - (pb), 1, 4 * (r_ - 3) + g_ - 2); \
    else if (r_ == 3 && g_ == 6) { dd = p0 - p1; PINS(dd); } \
    else if (r_ == 3 && g_ == 7) { da = sa == NAS - 1 ? -(NAS - 1) * A_BYTES : A_BYTES; PINS(da); } \
    else if (r_ == 4 && g_ == 6) { sa = sa == NAS - 1 ? 0 : sa + 1; PINS(sa); } \
    else if (r_ == 5 && g_ < 4) { offB[g_] += dd; PINV(offB[g_]); } \
    else if (r_ == 5 && g_ < 6) { offA[g_ - 4] += da; PINV(offA[g_ - 4]); } \
    else if (r_ == 6 && g_ < 4) RDA(g_ >> 1, g_ & 1); \
    else if (r_ == 6 && g_ == 4) { p1 += H_BYTES; PINS(p1); } \
    else if (r_ == 6 && g_ == 5) { p1 = p1 == re1 ? rb1 : p1; PINS(p1); } \
    else if (r_ == 6 && g_ == 6) { dd = p1 - p0; PINS(dd); } \
    else if (r_ == 7 && g_ < 2) RDA(2, g_); \
  } while (0)
#define R8(M, pb, i) M(pb, i); M(pb, (i) + 1); M(pb, (i) + 2); M(pb, (i) + 3); M(pb, (i) + 4); M(pb, (i) + 5); M(pb, (i) + 6); M(pb, (i) + 7)
#define MG(pb, i) do { MF(pb, i); GAP(pb, i); SB(); } while (0)
#define TILE(pb) do { \
    R8(MG, pb, 0); R8(MG, pb, 8); R8(MG, pb, 16); R8(MG, pb, 24); R8(MG, pb, 32); R8(MG, pb, 40); \
    LGKM0(); BAR(); SB();              \
    R8(MG, pb, 48); R8(MG, pb, 56); \
  } while (0)

#pragma unroll
  for (int m = 0; m < 8; ++m) asm volatile("" : "+v"(acc[m][0]), "+v"(acc[m][1]), "+v"(acc[m][2]), "+v"(acc[m][3]));
  BAR();
#pragma unroll
  for (int i = 0; i < 8; ++i) RDB(0, 0, i);
#pragma unroll
  for (int i = 0; i < 4; ++i) offB[i] += rb1 - rb0;
#pragma unroll
  for (int i = 0; i < 8; ++i) RDB(0, 1, i);
#pragma unroll
  for (int i = 0; i < 6; ++i) RDA(i >> 1, i & 1);
  int p0 = rb0 + H_BYTES, p1 = rb1 + H_BYTES, dd = p1 - p0, sa = 0, da = 0;
#pragma unroll
  for (int i = 0; i < 4; ++i) offB[i] += p0 - rb1;
  SB();
  for (int t = 0; t < NT - 2; t += 2) {
    TILE(0);
    TILE(1);
  }
  TILE(0);
  float mx0 = 0.f, mx1 = 0.f;
#define AMX(i) do { \
    if ((i) & 1) asm volatile("v_max3_f32 %0, |%1|, |%2|, %0" : "+v"(mx1) : "v"(acc[MI(i)][NI(i)][2]), "v"(acc[MI(i)][NI(i)][3])); \
    else asm volatile("v_max3_f32 %0, |%1|, |%2|, %0" : "+v"(mx0) : "v"(acc[MI(i)][NI(i)][0]), "v"(acc[MI(i)][NI(i)][1])); } while (0)
#define ML(pb, i) do { MF(pb, i); if (((i) >> 3) < 5 && ((i) & 7) < 2) RDA(((i) >> 3) + 3, (i) & 1); if ((i) >= 10) AMX(((i) - 10) & 63); SB(); } while (0)
  R8(ML, 1, 0); R8(ML, 1, 8); R8(ML, 1, 16); R8(ML, 1, 24); R8(ML, 1, 32); R8(ML, 1, 40); R8(ML, 1, 48); R8(ML, 1, 56);
#pragma unroll
  for (int i = 54; i < 64; ++i) { AMX(i); SB(); }
  const unsigned mw = wave_umax(max(__float_as_uint(mx0), __float_as_uint(mx1)));
  if (lane == 0) ctl[C_RED + wid] = mw;
  char* orow = p.out[dst] + (size_t)(p.rank * MLOC + m0 + wm * 128 + wn * 8 + (lane >> 3)) * NSH + ct * BN + (lane & 7) * 16;
  LGKM0(); BAR();
  {
    const char* gp = myws + WS_AMAX + ((size_t)(call & 1) * NR + 2 * wid) * NWG * 8 + lane * 16;
    const unsigned long long t0 = __builtin_amdgcn_s_memrealtime();
    unsigned m = 0; bool gave_up = false;
    __builtin_amdgcn_s_sleep(12);
    for (;;) {
      u32x4 a, c, d, e; unsigned tw;
      asm volatile("global_load_dwordx4 %0, %5, off sc0 sc1\n\t"
                   "global_load_dwordx4 %1, %5, off offset:1024 sc0 sc1\n\t"
                   "global_load_dwordx4 %2, %5, off offset:2048 sc0 sc1\n\t"
                   "global_load_dwordx4 %3, %5, off offset:3072 sc0 sc1\n\t"
                   "global_load_dword %4, %6, off sc1\n\t"
                   "s_waitcnt vmcnt(0)"
                   : "=&v"(a), "=&v"(c), "=&v"(d), "=&v"(e), "=&v"(tw) : "v"(gp), "v"(tmo) : "memory");
      const bool ok = a.y == call && a.w == call && c.y == call && c.w == call && d.y == call && d.w == call && e.y == call && e.w == call;
      if (RFL(tw) == call) { gave_up = true; break; }
      if (__all(ok)) { m = max(max(max(a.x, a.z), max(c.x, c.z)), max(max(d.x, d.z), max(e.x, e.z))); break; }
      if (__builtin_amdgcn_s_memrealtime() - t0 > DEADLINE) { __hip_atomic_store(tmo, call, __ATOMIC_RELAXED, __HIP_MEMORY_SCOPE_AGENT); gave_up = true; break; }
      __builtin_amdgcn_s_sleep(6);
    }
    m = wave_umax(m);
    if (lane == 0) { ctl[C_RED + 4 + wid] = m; ctl[C_BAD + wid] = gave_up ? 1u : 0u; }
  }
  LGKM0(); BAR();
  const float amax = __uint_as_float(max(max(ctl[C_RED + 4], ctl[C_RED + 5]), max(ctl[C_RED + 6], ctl[C_RED + 7])));
  const bool bad = (ctl[C_BAD] | ctl[C_BAD + 1] | ctl[C_BAD + 2] | ctl[C_BAD + 3]) != 0u;
  const float inv = 127.0f / amax;
  const float MAGIC = 12582912.0f;
  auto quant = [&](int m) {
    unsigned dw[4];
#pragma unroll
    for (int n = 0; n < 4; ++n) {
      unsigned q[4];
#pragma unroll
      for (int e = 0; e < 4; ++e)
        q[e] = __float_as_uint(__builtin_amdgcn_fmed3f(__builtin_fmaf(acc[m][n][e], inv, MAGIC), MAGIC - 127.0f, MAGIC + 127.0f));
      const unsigned lo = __builtin_amdgcn_perm(q[1], q[0], 0x0c0c0400u), hi = __builtin_amdgcn_perm(q[3], q[2], 0x04000c0cu);
      dw[n] = lo | hi;
    }
    auto s02 = __builtin_amdgcn_permlane32_swap(dw[0], dw[2], false, false);
    auto s13 = __builtin_amdgcn_permlane32_swap(dw[1], dw[3], false, false);
    auto s01 = __builtin_amdgcn_permlane16_swap(s02[0], s13[0], false, false);
    auto s23 = __builtin_amdgcn_permlane16_swap(s02[1], s13[1], false, false);
    u32x4 v = {s01[0], s01[1], s23[0], s23[1]};
    if (bad) v = (u32x4){0x80808080u, 0x80808080u, 0x80808080u, 0x80808080u};
    return v;
  };
  LAS char* xw = lds + LDS_XCH + (wm * 128 + fr) * 128 + (((wn * 4 + fq) ^ (fr & 7)) << 4);
  const LAS char* xr = lds + LDS_XCH + (wm * 128 + wn * 8 + (lane >> 3)) * 128 + (((lane & 7) ^ (lane >> 3)) << 4);
  *(LAS u32x4*)xw = quant(0);
  LGKM0(); BAR();
  store16_sys(orow, *(const LAS u32x4*)xr);
  SB();
#pragma unroll
  for (int m = 1; m < 8; ++m) *(LAS u32x4*)(xw + m * 2048) = quant(m);
  LGKM0(); BAR();
  u32x4 ov[8];
#pragma unroll
  for (int m = 1; m < 8; ++m) ov[m] = *(const LAS u32x4*)(xr + m * 2048);
#pragma unroll
  for (int m = 1; m < 8; ++m) {
    if (dst != p.rank) WAIT_V(0);
    store16_sys(orow + (size_t)m * 16 * NSH, ov[m]);
  }
  WAIT_V(0);
  BAR();
}
}

extern "C" void kernel_launch(int rank, int nranks, ncclComm_t comm, hipStream_t stream,
                              void* const* d_in, const int* in_sizes, int n_in,
                              void* d_out, int out_size, const int* in_dtypes, int out_dtype,
                              void* const* d_out_all, void* const* d_ws_all, size_t ws_bytes) {
  if (nranks != NR || (unsigned)rank >= (unsigned)NR || ws_bytes < WS_NEED || n_in != 2 || in_sizes[0] != MLOC * KD || in_sizes[1] != KD * ND ||
      out_size != NR * MLOC * NSH) { fprintf(stderr, "unsupported configuration\n"); return; }
  Params p;
  p.x = (const unsigned short*)d_in[0]; p.w = (const unsigned short*)d_in[1];
  for (int i = 0; i < NR; ++i) { p.out[i] = (char*)d_out_all[i]; p.ws[i] = (char*)d_ws_all[i]; }
  p.rank = rank; p.pad = 0;
  hipLaunchKernelGGL(gemm_a2a_q8, dim3(NWG), dim3(320), 0, stream, p);
  HIP_CHECK(hipGetLastError());
}
```

```cpp
#include <hip/hip_runtime.h>
#include <rccl/rccl.h>
#include <cstdio>
#include <cstdint>

#define HIP_CHECK(expr) do { hipError_t _e = (expr); if (_e != hipSuccess) { \
    fprintf(stderr, "HIP error %s at %s:%d\n", hipGetErrorString(_e), __FILE__, __LINE__); \
    return; } } while (0)

namespace {
#define LAS __attribute__((address_space(3)))
typedef short bf16x8 __attribute__((ext_vector_type(8)));
typedef short s16x4 __attribute__((ext_vector_type(4)));
typedef float f32x4 __attribute__((ext_vector_type(4)));
typedef unsigned u32x4 __attribute__((ext_vector_type(4)));

constexpr int NR = 8;
constexpr int MLOC = 512, KD = 4096, ND = 16384, NSH = ND / NR;
constexpr int BM = 256, BN = 128, BK = 64, NT = KD / BK;
constexpr int NWG = 256;
constexpr int A_BYTES = 32768, W_BYTES = 32768, NAS = 3, NS = KD / 128;
constexpr int LDS_A = 0, LDS_W = NAS * A_BYTES, LDS_BYTES = LDS_W + 2 * W_BYTES;
constexpr int LDS_CTL = LDS_A + A_BYTES;
constexpr int LDS_XCH = LDS_A + 2 * A_BYTES;
constexpr int C_RED = 0;
constexpr int C_BAD = 8;
constexpr size_t WS_CNT = 0;
constexpr size_t WS_TMO = 4096;
constexpr size_t WS_AMAX = 8192;
constexpr size_t WS_DONE = 65536;
constexpr size_t WS_NEED = 1u << 20;
constexpr unsigned long long DEADLINE = 50000000ull;

struct Params {
  const unsigned short* x; const unsigned short* w;
  char* out[NR]; char* ws[NR];
  int rank; int pad;
};

#define WAIT_V(n) asm volatile("s_waitcnt vmcnt(%0)" :: "i"(n) : "memory")
#define BAR() do { asm volatile("" ::: "memory"); __builtin_amdgcn_s_barrier(); asm volatile("" ::: "memory"); } while (0)
#define LGKM0() __builtin_amdgcn_s_waitcnt(0xC07F)
#define RFL(x) __builtin_amdgcn_readfirstlane(x)
#define SB() __builtin_amdgcn_sched_barrier(0)

__device__ __forceinline__ void store16_sys(void* ptr, u32x4 v) {
  asm volatile("global_store_dwordx4 %0, %1, off sc0 sc1\n\ts_nop 1" :: "v"(ptr), "v"(v) : "memory");
}
__device__ __forceinline__ unsigned wave_umax(unsigned v) {
  v = max(v, (unsigned)__builtin_amdgcn_update_dpp(0, (int)v, 0xB1, 0xF, 0xF, true));
  v = max(v, (unsigned)__builtin_amdgcn_update_dpp(0, (int)v, 0x4E, 0xF, 0xF, true));
  v = max(v, (unsigned)__builtin_amdgcn_update_dpp(0, (int)v, 0x141, 0xF, 0xF, true));
  v = max(v, (unsigned)__builtin_amdgcn_update_dpp(0, (int)v, 0x140, 0xF, 0xF, true));
  auto a = __builtin_amdgcn_permlane16_swap(v, v, false, false); v = max(a[0], a[1]);
  auto c = __builtin_amdgcn_permlane32_swap(v, v, false, false); return max(c[0], c[1]);
}
#define GL8(base, v, dst) \
    asm volatile("s_mov_b32 m0, %1\n\ts_nop 0\n\t" \
                 "global_load_lds_dwordx4 %2, %0 offset:-4096\n\tglobal_load_lds_dwordx4 %3, %0 offset:-3072\n\t" \
                 "global_load_lds_dwordx4 %4, %0 offset:-2048\n\tglobal_load_lds_dwordx4 %5, %0 offset:-1024\n\t" \
                 "global_load_lds_dwordx4 %6, %0\n\tglobal_load_lds_dwordx4 %7, %0 offset:1024\n\t" \
                 "global_load_lds_dwordx4 %8, %0 offset:2048\n\tglobal_load_lds_dwordx4 %9, %0 offset:3072" \
                 :: "s"(base), "s"((dst) + 4096), "v"((v)[0]), "v"((v)[1]), "v"((v)[2]), "v"((v)[3]), "v"((v)[4]), "v"((v)[5]), "v"((v)[6]), "v"((v)[7]) : "memory")

__global__ __launch_bounds__(320, 2) void gemm_a2a_q8(const Params p) {
  __shared__ __attribute__((aligned(16))) char smem[LDS_BYTES];
  LAS char* lds = (LAS char*)smem;
  LAS unsigned* ctl = (LAS unsigned*)(lds + LDS_CTL);
  const int tid = threadIdx.x, lane = tid & 63;
  const int wid = RFL(tid >> 6);
  const int b = blockIdx.x, xg = b & 7, j = b >> 3;
  const int dst = (xg + p.rank) & 7;
  const int ct = j >> 1, rt = j & 1, m0 = rt * BM;
  const int n0 = dst * NSH + ct * BN;
  const int s0 = xg * 4;
  char* const myws = p.ws[p.rank];
  unsigned* const cnt = (unsigned*)(myws + WS_CNT) + b;
  unsigned* const tmo = (unsigned*)(myws + WS_TMO);
#define LOAD_CALL(call) unsigned call; asm volatile("s_load_dword %0, %1, 0x0\n\ts_waitcnt lgkmcnt(0)" : "=s"(call) : "s"(cnt) : "memory"); call += 1

  if (wid == 4) {
    const unsigned ldsb = (unsigned)(uintptr_t)lds;
    unsigned va[8], vb[8];
#pragma unroll
    for (int i = 0; i < 8; ++i) {
      const int kr = 8 * (i >> 1) + (lane >> 3);
      vb[i] = (unsigned)(kr * (ND * 2) + (i & 1) * 128 + (((lane & 7) ^ ((((i >> 1) & 1) << 2) | ((lane >> 3) & 2))) << 4)) + 8192u - i * 1024u;
      const int row = 8 * i + (lane >> 3);
      va[i] = (unsigned)(row * (KD * 2) + (((lane & 7) ^ ((row >> 1) & 7)) << 4)) + 8192u - i * 1024u;
    }
    const char* abase = (const char*)p.x + (size_t)m0 * (KD * 2) - 4096;
    const char* wbase = (const char*)p.w + (size_t)n0 * 2 - 4096;
    auto issue_a = [&](int u, int slot) {
      const char* at = abase + (unsigned)((((u >> 1) + s0) & (NS - 1)) * 256 + (u & 1) * (64 * KD * 2));
#pragma unroll
      for (int g = 0; g < 4; ++g) GL8(at + (size_t)(g >> 1) * (128 * KD * 2) + (g & 1) * 128, va, ldsb + LDS_A + slot * A_BYTES + (g & 1) * 16384 + (g >> 1) * 8192);
    };
    auto issue_w = [&](int st, int h) {
      const char* bt = wbase + (unsigned)((((st + s0) & (NS - 1)) * 2 + h) * (64 * ND * 2));
      const unsigned d = ldsb + LDS_W + (st & 1) * W_BYTES + h * 16384;
      GL8(bt, vb, d); GL8(bt + 32 * ND * 2, vb, d + 8192);
    };
    for (int i = 0; i < (ct & 7); ++i) __builtin_amdgcn_s_sleep(2);
    issue_w(0, rt); issue_a(0, 0); issue_w(0, 1 - rt);
    LOAD_CALL(call);
    char* pw = p.ws[0];
#pragma unroll
    for (int i = 1; i < NR; ++i) pw = (lane & 7) == i ? p.ws[i] : pw;
    unsigned long long* gran = (unsigned long long*)(pw + WS_AMAX) + ((size_t)(call & 1) * NR + p.rank) * NWG + b;
    asm volatile("" : "+v"(gran));
    WAIT_V(0); BAR();
    issue_a(1, 1); issue_w(1, rt);
    WAIT_V(16); BAR();
    issue_a(2, 2); issue_w(1, 1 - rt);
    int sa = 0;
    for (int t = 1; t < NT - 1; ++t) {
      const int st = (t + 3) >> 1;
      if (st < NS) {
        issue_a(t + 2, sa); issue_w(st, (t & 1) ? rt : 1 - rt);
        if (t & 1) WAIT_V(48); else WAIT_V(63);
      } else if (t + 2 < NT) { issue_a(t + 2, sa); WAIT_V(32); } else WAIT_V(0);
      sa = sa == NAS - 1 ? 0 : sa + 1;
      BAR();
    }
    BAR();
    const unsigned m = max(max(ctl[C_RED], ctl[C_RED + 1]), max(ctl[C_RED + 2], ctl[C_RED + 3]));
    if (lane < NR) __hip_atomic_store(gran, ((unsigned long long)call << 32) | m, __ATOMIC_RELAXED, __HIP_MEMORY_SCOPE_SYSTEM);
    if (lane == 0) __hip_atomic_store(cnt, call, __ATOMIC_RELAXED, __HIP_MEMORY_SCOPE_AGENT);
    BAR();
    BAR(); BAR();
    BAR();
    if (lane == 0 && dst != p.rank)
      __hip_atomic_store((unsigned*)(p.ws[dst] + WS_DONE) + p.rank * 32 + j, call, __ATOMIC_RELAXED, __HIP_MEMORY_SCOPE_SYSTEM);
    if (b == 0) {
      const char* fp = myws + WS_DONE + lane * 16;
      const bool mine = (lane >> 3) == p.rank;
      const unsigned long long t0 = __builtin_amdgcn_s_memrealtime();
      for (;;) {
        u32x4 a; unsigned tw;
        asm volatile("global_load_dwordx4 %0, %2, off sc0 sc1\n\tglobal_load_dword %1, %3, off sc1\n\ts_waitcnt vmcnt(0)" : "=&v"(a), "=&v"(tw) : "v"(fp), "v"(tmo) : "memory");
        const bool ok = mine || (a.x >= call && a.y >= call && a.z >= call && a.w >= call);
        if (__all(ok)) break;
        if (RFL(tw) == call || __builtin_amdgcn_s_memrealtime() - t0 > DEADLINE) {
          __hip_atomic_store(tmo, call, __ATOMIC_RELAXED, __HIP_MEMORY_SCOPE_AGENT);
          if (lane == 0) p.out[p.rank][(size_t)(p.rank * MLOC) * NSH] = (char)-128;
          break;
        }
        __builtin_amdgcn_s_sleep(1);
      }
    }
    return;
  }

  const int wm = wid >> 1, wn = wid & 1, fr = lane & 15, fq = lane >> 4;
  LOAD_CALL(call);
  f32x4 acc[8][4];
#pragma unroll
  for (int m = 0; m < 8; ++m)
#pragma unroll
    for (int n = 0; n < 4; ++n) acc[m][n] = (f32x4){0.f, 0.f, 0.f, 0.f};
  int offA[2], offB[4];
#pragma unroll
  for (int ks = 0; ks < 2; ++ks) offA[ks] = LDS_A + (wm * 64 + fr) * 128 + (((4 * ks + fq) ^ ((fr >> 1) & 7)) << 4);
  {
    const int q = (lane & 15) >> 2, pp = lane & 3;
#pragma unroll
    for (int n = 0; n < 4; ++n)
      offB[n] = LDS_W + 2048 * fq + 1024 * wn + 128 * q + (((2 * n + (pp >> 1)) ^ (((fq & 1) << 2) | (q & 2))) << 4) + 8 * (pp & 1);
  }
  bf16x8 Af[2][4];
  s16x4 Bl[4][4], Bh[4][4];
#define RDA(m, ks) Af[(m) & 1][ks] = *(const LAS bf16x8*)(lds + offA[(ks) & 1] + ((ks) >> 1) * 16384 + ((m) & 3) * 2048)
#define RDB(n, ks) do { \
    Bl[n][ks] = __builtin_amdgcn_ds_read_tr16_b64_v4i16((LAS s16x4*)(lds + offB[n] + (ks) * 8192)); \
    Bh[n][ks] = __builtin_amdgcn_ds_read_tr16_b64_v4i16((LAS s16x4*)(lds + offB[n] + (ks) * 8192 + 512)); } while (0)
#define MI(i) ((i) >> 4)
#define NI(i) (((((i) >> 4) & 1) && ((i) >> 4) != 7) ? 3 - (((i) >> 2) & 3) : (((i) >> 2) & 3))
#define KI(i) ((((i) >> 2) & 1) ? 3 - ((i) & 3) : ((i) & 3))
#define MF(i) asm volatile("v_mfma_f32_16x16x32_bf16 %0, %1, %2, %0" : "+v"(acc[MI(i)][NI(i)]) \
    : "v"(__builtin_shufflevector(Bl[NI(i)][KI(i)], Bh[NI(i)][KI(i)], 0, 1, 2, 3, 4, 5, 6, 7)), "v"(Af[MI(i) & 1][KI(i)]))
#define PINV(x) asm volatile("" : "+v"(x))
#define PINS(x) asm volatile("" : "+s"(x))
#define GAP(i) do { \
    constexpr int r_ = (i) >> 4, g_ = (i) & 15; \
    if (r_ == 0 && g_ < 4) RDB(3, 3 - g_); \
    else if (r_ == 0 && g_ < 8) RDA(1, g_ - 4); \
    else if (r_ > 0 && g_ < 4) RDA(r_ + 1, g_); \
    else if (r_ == 7) RDB((g_ >> 2) - 1, KI(g_ - 4)); \
    else if (r_ == 1 && g_ < 8) { offB[g_ - 4] += dw; PINV(offB[g_ - 4]); } \
    else if (r_ == 1 && g_ == 8) { dw = -dw; PINS(dw); } \
    else if ((r_ == 2 || r_ == 6) && g_ == 4) { da = sa == NAS - 1 ? -(NAS - 1) * A_BYTES : A_BYTES; PINS(da); } \
    else if ((r_ == 2 || r_ == 6) && g_ == 5) { sa = sa == NAS - 1 ? 0 : sa + 1; PINS(sa); } \
    else if ((r_ == 2 || r_ == 6) && g_ < 8) { offA[g_ - 6] += da; PINV(offA[g_ - 6]); } \
  } while (0)
#define R4(M, i) M(i); M((i) + 1); M((i) + 2); M((i) + 3)
#define R16(M, i) R4(M, i); R4(M, (i) + 4); R4(M, (i) + 8); R4(M, (i) + 12)
#define MG(i) do { MF(i); GAP(i); SB(); } while (0)
#define SYNC() do { LGKM0(); BAR(); SB(); } while (0)

#pragma unroll
  for (int m = 0; m < 8; ++m) asm volatile("" : "+v"(acc[m][0]), "+v"(acc[m][1]), "+v"(acc[m][2]), "+v"(acc[m][3]));
  int sa = 0, da = 0, dw = W_BYTES;
  BAR();
#pragma unroll
  for (int i = 0; i < 4; ++i) RDA(0, i);
#pragma unroll
  for (int i = 0; i < 12; ++i) RDB(i >> 2, i & 3);
  SB();
  for (int st = 0; st < NS - 1; ++st) {
    R16(MG, 0); R16(MG, 16); R16(MG, 32);
    SYNC();
    R16(MG, 48); R16(MG, 64); R16(MG, 80); R16(MG, 96);
    SYNC();
    R16(MG, 112);
  }
  float mx0 = 0.f, mx1 = 0.f;
#define AMX(i) do { \
    if ((i) & 2) asm volatile("v_max3_f32 %0, |%1|, |%2|, %0" : "+v"(mx1) : "v"(acc[MI(i)][NI(i)][2]), "v"(acc[MI(i)][NI(i)][3])); \
    else asm volatile("v_max3_f32 %0, |%1|, |%2|, %0" : "+v"(mx0) : "v"(acc[MI(i)][NI(i)][0]), "v"(acc[MI(i)][NI(i)][1])); } while (0)
#define ML(i) do { MF(i); if ((i) < 112) GAP((i) % 112); if ((i) >= 20 && !((i) & 1)) AMX(((i) - 20) & 127); SB(); } while (0)
  R16(ML, 0); R16(ML, 16); R16(ML, 32);
  SYNC();
  R16(ML, 48); R16(ML, 64); R16(ML, 80); R16(ML, 96); R16(ML, 112);
#pragma unroll
  for (int i = 108; i < 128; i += 2) { AMX(i); SB(); }
  const unsigned mw = wave_umax(max(__float_as_uint(mx0), __float_as_uint(mx1)));
  if (lane == 0) ctl[C_RED + wid] = mw;
  char* orow = p.out[dst] + (size_t)(p.rank * MLOC + m0 + wm * 128 + wn * 8 + (lane >> 3)) * NSH + ct * BN + (lane & 7) * 16;
  LGKM0(); BAR();
  {
    const char* gp = myws + WS_AMAX + ((size_t)(call & 1) * NR + 2 * wid) * NWG * 8 + lane * 16;
    const unsigned long long t0 = __builtin_amdgcn_s_memrealtime();
    unsigned m = 0; bool gave_up = false;
    __builtin_amdgcn_s_sleep(12);
    for (;;) {
      u32x4 a, c, d, e; unsigned tw;
      asm volatile("global_load_dwordx4 %0, %5, off sc0 sc1\n\t"
                   "global_load_dwordx4 %1, %5, off offset:1024 sc0 sc1\n\t"
                   "global_load_dwordx4 %2, %5, off offset:2048 sc0 sc1\n\t"
                   "global_load_dwordx4 %3, %5, off offset:3072 sc0 sc1\n\t"
                   "global_load_dword %4, %6, off sc1\n\t"
                   "s_waitcnt vmcnt(0)"
                   : "=&v"(a), "=&v"(c), "=&v"(d), "=&v"(e), "=&v"(tw) : "v"(gp), "v"(tmo) : "memory");
      const bool ok = a.y == call && a.w == call && c.y == call && c.w == call && d.y == call && d.w == call && e.y == call && e.w == call;
      if (RFL(tw) == call) { gave_up = true; break; }
      if (__all(ok)) { m = max(max(max(a.x, a.z), max(c.x, c.z)), max(max(d.x, d.z), max(e.x, e.z))); break; }
      if (__builtin_amdgcn_s_memrealtime() - t0 > DEADLINE) { __hip_atomic_store(tmo, call, __ATOMIC_RELAXED, __HIP_MEMORY_SCOPE_AGENT); gave_up = true; break; }
      __builtin_amdgcn_s_sleep(6);
    }
    m = wave_umax(m);
    if (lane == 0) { ctl[C_RED + 4 + wid] = m; ctl[C_BAD + wid] = gave_up ? 1u : 0u; }
  }
  LGKM0(); BAR();
  const float amax = __uint_as_float(max(max(ctl[C_RED + 4], ctl[C_RED + 5]), max(ctl[C_RED + 6], ctl[C_RED + 7])));
  const bool bad = (ctl[C_BAD] | ctl[C_BAD + 1] | ctl[C_BAD + 2] | ctl[C_BAD + 3]) != 0u;
  const float inv = 127.0f / amax;
  const float MAGIC = 12582912.0f;
  auto quant = [&](int m) {
    unsigned dw[4];
#pragma unroll
    for (int n = 0; n < 4; ++n) {
      unsigned q[4];
#pragma unroll
      for (int e = 0; e < 4; ++e)
        q[e] = __float_as_uint(__builtin_amdgcn_fmed3f(__builtin_fmaf(acc[m][n][e], inv, MAGIC), MAGIC - 127.0f, MAGIC + 127.0f));
      const unsigned lo = __builtin_amdgcn_perm(q[1], q[0], 0x0c0c0400u), hi = __builtin_amdgcn_perm(q[3], q[2], 0x04000c0cu);
      dw[n] = lo | hi;
    }
    auto s02 = __builtin_amdgcn_permlane32_swap(dw[0], dw[2], false, false);
    auto s13 = __builtin_amdgcn_permlane32_swap(dw[1], dw[3], false, false);
    auto s01 = __builtin_amdgcn_permlane16_swap(s02[0], s13[0], false, false);
    auto s23 = __builtin_amdgcn_permlane16_swap(s02[1], s13[1], false, false);
    u32x4 v = {s01[0], s01[1], s23[0], s23[1]};
    if (bad) v = (u32x4){0x80808080u, 0x80808080u, 0x80808080u, 0x80808080u};
    return v;
  };
  LAS char* xw = lds + LDS_XCH + (wm * 128 + fr) * 128 + (((wn * 4 + fq) ^ (fr & 7)) << 4);
  const LAS char* xr = lds + LDS_XCH + (wm * 128 + wn * 8 + (lane >> 3)) * 128 + (((lane & 7) ^ (lane >> 3)) << 4);
  *(LAS u32x4*)xw = quant(0);
  LGKM0(); BAR();
  store16_sys(orow, *(const LAS u32x4*)xr);
  SB();
#pragma unroll
  for (int m = 1; m < 8; ++m) *(LAS u32x4*)(xw + m * 2048) = quant(m);
  LGKM0(); BAR();
  u32x4 ov[8];
#pragma unroll
  for (int m = 1; m < 8; ++m) ov[m] = *(const LAS u32x4*)(xr + m * 2048);
#pragma unroll
  for (int m = 1; m < 8; ++m) {
    if (dst != p.rank) WAIT_V(0);
    store16_sys(orow + (size_t)m * 16 * NSH, ov[m]);
  }
  WAIT_V(0);
  BAR();
}
}

extern "C" void kernel_launch(int rank, int nranks, ncclComm_t comm, hipStream_t stream,
                              void* const* d_in, const int* in_sizes, int n_in,
                              void* d_out, int out_size, const int* in_dtypes, int out_dtype,
                              void* const* d_out_all, void* const* d_ws_all, size_t ws_bytes) {
  if (nranks != NR || (unsigned)rank >= (unsigned)NR || ws_bytes < WS_NEED || n_in != 2 || in_sizes[0] != MLOC * KD || in_sizes[1] != KD * ND ||
      out_size != NR * MLOC * NSH) { fprintf(stderr, "unsupported configuration\n"); return; }
  Params p;
  p.x = (const unsigned short*)d_in[0]; p.w = (const unsigned short*)d_in[1];
  for (int i = 0; i < NR; ++i) { p.out[i] = (char*)d_out_all[i]; p.ws[i] = (char*)d_ws_all[i]; }
  p.rank = rank; p.pad = 0;
  hipLaunchKernelGGL(gemm_a2a_q8, dim3(NWG), dim3(320), 0, stream, p);
  HIP_CHECK(hipGetLastError());
}
```

```cpp
#include <hip/hip_runtime.h>
#include <rccl/rccl.h>
#include <cstdio>
#include <cstdint>

#define HIP_CHECK(expr) do { hipError_t _e = (expr); if (_e != hipSuccess) { \
    fprintf(stderr, "HIP error %s at %s:%d\n", hipGetErrorString(_e), __FILE__, __LINE__); \
    return; } } while (0)

namespace {
#define LAS __attribute__((address_space(3)))
typedef short bf16x8 __attribute__((ext_vector_type(8)));
typedef short s16x4 __attribute__((ext_vector_type(4)));
typedef float f32x4 __attribute__((ext_vector_type(4)));
typedef unsigned u32x4 __attribute__((ext_vector_type(4)));

constexpr int NR = 8;
constexpr int MLOC = 512, KD = 4096, ND = 16384, NSH = ND / NR;
constexpr int BM = 256, BN = 128, BK = 64, NT = KD / BK;
constexpr int NWG = 256;
constexpr int A_BYTES = 32768, W_BYTES = 32768, NAS = 3, NS = KD / 128;
constexpr int LDS_A = 0, LDS_W = NAS * A_BYTES, LDS_BYTES = LDS_W + 2 * W_BYTES;
constexpr int LDS_CTL = LDS_A + A_BYTES;
constexpr int LDS_XCH = LDS_A + 2 * A_BYTES;
constexpr int C_RED = 0;
constexpr int C_BAD = 8;
constexpr size_t WS_CNT = 0;
constexpr size_t WS_TMO = 4096;
constexpr size_t WS_AMAX = 8192;
constexpr size_t WS_DONE = 65536;
constexpr size_t WS_NEED = 1u << 20;
constexpr int PACE_A = 12, PACE_B = 52, PACE_END_NS = 49500;
constexpr unsigned long long DEADLINE = 50000000ull;

struct Params {
  const unsigned short* x; const unsigned short* w;
  char* out[NR]; char* ws[NR];
  int rank; int pad;
};

#define WAIT_V(n) asm volatile("s_waitcnt vmcnt(%0)" :: "i"(n) : "memory")
#define BAR() do { asm volatile("" ::: "memory"); __builtin_amdgcn_s_barrier(); asm volatile("" ::: "memory"); } while (0)
#define LGKM0() __builtin_amdgcn_s_waitcnt(0xC07F)
#define RFL(x) __builtin_amdgcn_readfirstlane(x)
#define SB() __builtin_amdgcn_sched_barrier(0)

__device__ __forceinline__ void store16_sys(void* ptr, u32x4 v) {
  asm volatile("global_store_dwordx4 %0, %1, off sc0 sc1\n\ts_nop 1" :: "v"(ptr), "v"(v) : "memory");
}
__device__ __forceinline__ unsigned wave_umax(unsigned v) {
  v = max(v, (unsigned)__builtin_amdgcn_update_dpp(0, (int)v, 0xB1, 0xF, 0xF, true));
  v = max(v, (unsigned)__builtin_amdgcn_update_dpp(0, (int)v, 0x4E, 0xF, 0xF, true));
  v = max(v, (unsigned)__builtin_amdgcn_update_dpp(0, (int)v, 0x141, 0xF, 0xF, true));
  v = max(v, (unsigned)__builtin_amdgcn_update_dpp(0, (int)v, 0x140, 0xF, 0xF, true));
  auto a = __builtin_amdgcn_permlane16_swap(v, v, false, false); v = max(a[0], a[1]);
  auto c = __builtin_amdgcn_permlane32_swap(v, v, false, false); return max(c[0], c[1]);
}
#define GL8(base, v, dst) \
    asm volatile("s_mov_b32 m0, %1\n\ts_nop 0\n\t" \
                 "global_load_lds_dwordx4 %2, %0 offset:-4096\n\tglobal_load_lds_dwordx4 %3, %0 offset:-3072\n\t" \
                 "global_load_lds_dwordx4 %4, %0 offset:-2048\n\tglobal_load_lds_dwordx4 %5, %0 offset:-1024\n\t" \
                 "global_load_lds_dwordx4 %6, %0\n\tglobal_load_lds_dwordx4 %7, %0 offset:1024\n\t" \
                 "global_load_lds_dwordx4 %8, %0 offset:2048\n\tglobal_load_lds_dwordx4 %9, %0 offset:3072" \
                 :: "s"(base), "s"((dst) + 4096), "v"((v)[0]), "v"((v)[1]), "v"((v)[2]), "v"((v)[3]), "v"((v)[4]), "v"((v)[5]), "v"((v)[6]), "v"((v)[7]) : "memory")

__global__ __launch_bounds__(320, 2) void gemm_a2a_q8(const Params p) {
  __shared__ __attribute__((aligned(16))) char smem[LDS_BYTES];
  LAS char* lds = (LAS char*)smem;
  LAS unsigned* ctl = (LAS unsigned*)(lds + LDS_CTL);
  const int tid = threadIdx.x, lane = tid & 63;
  const int wid = RFL(tid >> 6);
  const int b = blockIdx.x, xg = b & 7, j = b >> 3;
  const int dst = (xg + p.rank) & 7;
  const int ct = j >> 1, rt = j & 1, m0 = rt * BM;
  const int n0 = dst * NSH + ct * BN;
  const int s0 = xg * 4;
  char* const myws = p.ws[p.rank];
  unsigned* const cnt = (unsigned*)(myws + WS_CNT) + b;
  unsigned* const tmo = (unsigned*)(myws + WS_TMO);
#define LOAD_CALL(call) unsigned call; asm volatile("s_load_dword %0, %1, 0x0\n\ts_waitcnt lgkmcnt(0)" : "=s"(call) : "s"(cnt) : "memory"); call += 1

  if (wid == 4) {
    const unsigned e0 = (unsigned)__builtin_amdgcn_s_memrealtime();
    const unsigned ldsb = (unsigned)(uintptr_t)lds;
    unsigned va[8], vb[8];
#pragma unroll
    for (int i = 0; i < 8; ++i) {
      const int kr = 8 * (i >> 1) + (lane >> 3);
      vb[i] = (unsigned)(kr * (ND * 2) + (i & 1) * 128 + (((lane & 7) ^ ((((i >> 1) & 1) << 2) | ((lane >> 3) & 2))) << 4)) + 8192u - i * 1024u;
      const int row = 8 * i + (lane >> 3);
      va[i] = (unsigned)(row * (KD * 2) + (((lane & 7) ^ ((row >> 1) & 7)) << 4)) + 8192u - i * 1024u;
    }
    const char* abase = (const char*)p.x + (size_t)m0 * (KD * 2) - 4096;
    const char* wbase = (const char*)p.w + (size_t)n0 * 2 - 4096;
    auto issue_a = [&](int u, int slot) {
      const char* at = abase + (unsigned)((((u >> 1) + s0) & (NS - 1)) * 256 + (u & 1) * (64 * KD * 2));
#pragma unroll
      for (int g = 0; g < 4; ++g) GL8(at + (size_t)(g >> 1) * (128 * KD * 2) + (g & 1) * 128, va, ldsb + LDS_A + slot * A_BYTES + (g & 1) * 16384 + (g >> 1) * 8192);
    };
    auto issue_w = [&](int st, int h) {
      const char* bt = wbase + (unsigned)((((st + s0) & (NS - 1)) * 2 + h) * (64 * ND * 2));
      const unsigned d = ldsb + LDS_W + (st & 1) * W_BYTES + h * 16384;
      GL8(bt, vb, d); GL8(bt + 32 * ND * 2, vb, d + 8192);
    };
    for (int i = 0; i < (ct & 7); ++i) __builtin_amdgcn_s_sleep(2);
    issue_w(0, rt); issue_a(0, 0); issue_w(0, 1 - rt);
    {
      unsigned vx[8];
#pragma unroll
      for (int i = 0; i < 8; ++i) vx[i] = i * (KD * 2) + lane * 16 + 8192u - i * 1024u;
      const char* xt = abase + (size_t)(16 * ct) * (KD * 2) + s0 * 256;
      GL8(xt, vx, ldsb + LDS_A + 2 * A_BYTES); GL8(xt + 8 * KD * 2, vx, ldsb + LDS_A + 2 * A_BYTES + 8192);
    }
    LOAD_CALL(call);
    char* pw = p.ws[0];
#pragma unroll
    for (int i = 1; i < NR; ++i) pw = (lane & 7) == i ? p.ws[i] : pw;
    unsigned long long* gran = (unsigned long long*)(pw + WS_AMAX) + ((size_t)(call & 1) * NR + p.rank) * NWG + b;
    asm volatile("" : "+v"(gran));
    WAIT_V(16); BAR();
    issue_a(1, 1); issue_w(1, rt);
    WAIT_V(16); BAR();
    issue_a(2, 2); issue_w(1, 1 - rt);
    int sa = 0;
    unsigned e4 = 0, ea = 0; int pace = 0;
    for (int t = 1; t < NT - 1; ++t) {
      const int st = (t + 3) >> 1;
      if (st < NS) {
        issue_a(t + 2, sa); issue_w(st, (t & 1) ? rt : 1 - rt);
        if (t & 1) WAIT_V(48); else WAIT_V(63);
      } else if (t + 2 < NT) { issue_a(t + 2, sa); WAIT_V(32); } else WAIT_V(0);
      sa = sa == NAS - 1 ? 0 : sa + 1;
      if (t == 4) e4 = (unsigned)__builtin_amdgcn_s_memrealtime();
      if (t == PACE_A) {
        ea = (unsigned)__builtin_amdgcn_s_memrealtime();
        const int rest = PACE_END_NS - (int)(ea - e0) * 10 - (int)(ea - e4) * 103 * (257 - 4 * PACE_B) / (40 * (PACE_A - 4));
        pace = min(max(rest, 0) / (PACE_B - 1 - PACE_A), 850);
      }
      if (t > PACE_A && t < PACE_B)
        while (((unsigned)__builtin_amdgcn_s_memrealtime() - ea) * 10u < (unsigned)((t - PACE_A) * pace)) __builtin_amdgcn_s_sleep(1);
      BAR();
    }
    BAR();
    const unsigned m = max(max(ctl[C_RED], ctl[C_RED + 1]), max(ctl[C_RED + 2], ctl[C_RED + 3]));
    if (lane < NR) asm volatile("global_store_dwordx2 %0, %1, off sc0 sc1" :: "v"(gran), "v"(((unsigned long long)call << 32) | m) : "memory");
    if (lane == 0) __hip_atomic_store(cnt, call, __ATOMIC_RELAXED, __HIP_MEMORY_SCOPE_AGENT);
    BAR();
    BAR(); BAR();
    BAR();
    if (lane == 0 && dst != p.rank)
      __hip_atomic_store((unsigned*)(p.ws[dst] + WS_DONE) + p.rank * 32 + j, call, __ATOMIC_RELAXED, __HIP_MEMORY_SCOPE_SYSTEM);
    if (b == 0) {
      const char* fp = myws + WS_DONE + lane * 16;
      const bool mine = (lane >> 3) == p.rank;
      const unsigned long long t0 = __builtin_amdgcn_s_memrealtime();
      for (;;) {
        u32x4 a; unsigned tw;
        asm volatile("global_load_dwordx4 %0, %2, off sc0 sc1\n\tglobal_load_dword %1, %3, off sc1\n\ts_waitcnt vmcnt(0)" : "=&v"(a), "=&v"(tw) : "v"(fp), "v"(tmo) : "memory");
        const bool ok = mine || (a.x >= call && a.y >= call && a.z >= call && a.w >= call);
        if (__all(ok)) break;
        if (RFL(tw) == call || __builtin_amdgcn_s_memrealtime() - t0 > DEADLINE) {
          __hip_atomic_store(tmo, call, __ATOMIC_RELAXED, __HIP_MEMORY_SCOPE_AGENT);
          if (lane == 0) p.out[p.rank][(size_t)(p.rank * MLOC) * NSH] = (char)-128;
          break;
        }
        __builtin_amdgcn_s_sleep(1);
      }
    }
    return;
  }

  const int wm = wid >> 1, wn = wid & 1, fr = lane & 15, fq = lane >> 4;
  LOAD_CALL(call);
  f32x4 acc[8][4];
#pragma unroll
  for (int m = 0; m < 8; ++m)
#pragma unroll
    for (int n = 0; n < 4; ++n) acc[m][n] = (f32x4){0.f, 0.f, 0.f, 0.f};
  int offA[2], offB[4];
#pragma unroll
  for (int ks = 0; ks < 2; ++ks) offA[ks] = LDS_A + (wm * 64 + fr) * 128 + (((4 * ks + fq) ^ ((fr >> 1) & 7)) << 4);
  {
    const int q = (lane & 15) >> 2, pp = lane & 3;
#pragma unroll
    for (int n = 0; n < 4; ++n)
      offB[n] = LDS_W + 2048 * fq + 1024 * wn + 128 * q + (((2 * n + (pp >> 1)) ^ (((fq & 1) << 2) | (q & 2))) << 4) + 8 * (pp & 1);
  }
  bf16x8 Af[2][4];
  s16x4 Bl[4][4], Bh[4][4];
#define RDA(m, ks) Af[(m) & 1][ks] = *(const LAS bf16x8*)(lds + offA[(ks) & 1] + ((ks) >> 1) * 16384 + ((m) & 3) * 2048)
#define RDB(n, ks) do { \
    Bl[n][ks] = __builtin_amdgcn_ds_read_tr16_b64_v4i16((LAS s16x4*)(lds + offB[n] + (ks) * 8192)); \
    Bh[n][ks] = __builtin_amdgcn_ds_read_tr16_b64_v4i16((LAS s16x4*)(lds + offB[n] + (ks) * 8192 + 512)); } while (0)
#define MI(i) ((i) >> 4)
#define NI(i) (((((i) >> 4) & 1) && ((i) >> 4) != 7) ? 3 - (((i) >> 2) & 3) : (((i) >> 2) & 3))
#define KI(i) ((((i) >> 2) & 1) ? 3 - ((i) & 3) : ((i) & 3))
#define MF(i) asm volatile("v_mfma_f32_16x16x32_bf16 %0, %1, %2, %0" : "+v"(acc[MI(i)][NI(i)]) \
    : "v"(__builtin_shufflevector(Bl[NI(i)][KI(i)], Bh[NI(i)][KI(i)], 0, 1, 2, 3, 4, 5, 6, 7)), "v"(Af[MI(i) & 1][KI(i)]))
#define PINV(x) asm volatile("" : "+v"(x))
#define PINS(x) asm volatile("" : "+s"(x))
#define GAP(i) do { \
    constexpr int r_ = (i) >> 4, g_ = (i) & 15; \
    if (r_ == 0 && g_ < 4) RDB(3, 3 - g_); \
    else if (r_ == 0 && g_ < 8) RDA(1, g_ - 4); \
    else if (r_ > 0 && g_ < 4) RDA(r_ + 1, g_); \
    else if (r_ == 7) RDB((g_ >> 2) - 1, KI(g_ - 4)); \
    else if (r_ == 1 && g_ < 8) { offB[g_ - 4] += dw; PINV(offB[g_ - 4]); } \
    else if (r_ == 1 && g_ == 8) { dw = -dw; PINS(dw); } \
    else if ((r_ == 2 || r_ == 6) && g_ == 4) { da = sa == NAS - 1 ? -(NAS - 1) * A_BYTES : A_BYTES; PINS(da); } \
    else if ((r_ == 2 || r_ == 6) && g_ == 5) { sa = sa == NAS - 1 ? 0 : sa + 1; PINS(sa); } \
    else if ((r_ == 2 || r_ == 6) && g_ < 8) { offA[g_ - 6] += da; PINV(offA[g_ - 6]); } \
  } while (0)
#define R4(M, i) M(i); M((i) + 1); M((i) + 2); M((i) + 3)
#define R16(M, i) R4(M, i); R4(M, (i) + 4); R4(M, (i) + 8); R4(M, (i) + 12)
#define MG(i) do { MF(i); GAP(i); SB(); } while (0)
#define SYNC() do { LGKM0(); BAR(); SB(); } while (0)

#pragma unroll
  for (int m = 0; m < 8; ++m) asm volatile("" : "+v"(acc[m][0]), "+v"(acc[m][1]), "+v"(acc[m][2]), "+v"(acc[m][3]));
  int sa = 0, da = 0, dw = W_BYTES;
  BAR();
#pragma unroll
  for (int i = 0; i < 4; ++i) RDA(0, i);
#pragma unroll
  for (int i = 0; i < 12; ++i) RDB(i >> 2, i & 3);
  SB();
  for (int st = 0; st < NS - 1; ++st) {
    R16(MG, 0); R16(MG, 16); R16(MG, 32);
    SYNC();
    R16(MG, 48); R16(MG, 64); R16(MG, 80); R16(MG, 96);
    SYNC();
    R16(MG, 112);
  }
  float mx0 = 0.f, mx1 = 0.f;
#define AMX(i) do { \
    if ((i) & 2) asm volatile("v_max3_f32 %0, |%1|, |%2|, %0" : "+v"(mx1) : "v"(acc[MI(i)][NI(i)][2]), "v"(acc[MI(i)][NI(i)][3])); \
    else asm volatile("v_max3_f32 %0, |%1|, |%2|, %0" : "+v"(mx0) : "v"(acc[MI(i)][NI(i)][0]), "v"(acc[MI(i)][NI(i)][1])); } while (0)
#define ML(i) do { MF(i); if ((i) < 112) GAP((i) % 112); if ((i) >= 20 && !((i) & 1)) AMX(((i) - 20) & 127); SB(); } while (0)
  R16(ML, 0); R16(ML, 16); R16(ML, 32);
  SYNC();
  R16(ML, 48); R16(ML, 64); R16(ML, 80); R16(ML, 96); R16(ML, 112);
  asm volatile("s_nop 1");
#pragma unroll
  for (int i = 108; i < 128; i += 2) { AMX(i); SB(); }
  const unsigned mw = wave_umax(max(__float_as_uint(mx0), __float_as_uint(mx1)));
  if (lane == 0) ctl[C_RED + wid] = mw;
  char* orow = p.out[dst] + (size_t)(p.rank * MLOC + m0 + wm * 128 + wn * 8 + (lane >> 3)) * NSH + ct * BN + (lane & 7) * 16;
  LGKM0(); BAR();
  {
    const char* gp = myws + WS_AMAX + ((size_t)(call & 1) * NR + 2 * wid) * NWG * 8 + lane * 16;
    const unsigned long long t0 = __builtin_amdgcn_s_memrealtime();
    unsigned m = 0; bool gave_up = false;
    __builtin_amdgcn_s_sleep(28);
    for (;;) {
      u32x4 a, c, d, e; unsigned tw;
      asm volatile("global_load_dwordx4 %0, %5, off sc0 sc1\n\t"
                   "global_load_dwordx4 %1, %5, off offset:1024 sc0 sc1\n\t"
                   "global_load_dwordx4 %2, %5, off offset:2048 sc0 sc1\n\t"
                   "global_load_dwordx4 %3, %5, off offset:3072 sc0 sc1\n\t"
                   "global_load_dword %4, %6, off sc1\n\t"
                   "s_waitcnt vmcnt(0)"
                   : "=&v"(a), "=&v"(c), "=&v"(d), "=&v"(e), "=&v"(tw) : "v"(gp), "v"(tmo) : "memory");
      const bool ok = a.y == call && a.w == call && c.y == call && c.w == call && d.y == call && d.w == call && e.y == call && e.w == call;
      if (RFL(tw) == call) { gave_up = true; break; }
      if (__all(ok)) { m = max(max(max(a.x, a.z), max(c.x, c.z)), max(max(d.x, d.z), max(e.x, e.z))); break; }
      if (__builtin_amdgcn_s_memrealtime() - t0 > DEADLINE) { __hip_atomic_store(tmo, call, __ATOMIC_RELAXED, __HIP_MEMORY_SCOPE_AGENT); gave_up = true; break; }
      __builtin_amdgcn_s_sleep(4);
    }
    m = wave_umax(m);
    if (lane == 0) { ctl[C_RED + 4 + wid] = m; ctl[C_BAD + wid] = gave_up ? 1u : 0u; }
  }
  LGKM0(); BAR();
  const float amax = __uint_as_float(max(max(ctl[C_RED + 4], ctl[C_RED + 5]), max(ctl[C_RED + 6], ctl[C_RED + 7])));
  const bool bad = (ctl[C_BAD] | ctl[C_BAD + 1] | ctl[C_BAD + 2] | ctl[C_BAD + 3]) != 0u;
  const float inv = 127.0f / amax;
  const float MAGIC = 12582912.0f;
  auto quant = [&](int m) {
    unsigned dw[4];
#pragma unroll
    for (int n = 0; n < 4; ++n) {
      unsigned q[4];
#pragma unroll
      for (int e = 0; e < 4; ++e)
        q[e] = __float_as_uint(__builtin_amdgcn_fmed3f(__builtin_fmaf(acc[m][n][e], inv, MAGIC), MAGIC - 127.0f, MAGIC + 127.0f));
      const unsigned lo = __builtin_amdgcn_perm(q[1], q[0], 0x0c0c0400u), hi = __builtin_amdgcn_perm(q[3], q[2], 0x04000c0cu);
      dw[n] = lo | hi;
    }
    auto s02 = __builtin_amdgcn_permlane32_swap(dw[0], dw[2], false, false);
    auto s13 = __builtin_amdgcn_permlane32_swap(dw[1], dw[3], false, false);
    auto s01 = __builtin_amdgcn_permlane16_swap(s02[0], s13[0], false, false);
    auto s23 = __builtin_amdgcn_permlane16_swap(s02[1], s13[1], false, false);
    u32x4 v = {s01[0], s01[1], s23[0], s23[1]};
    if (bad) v = (u32x4){0x80808080u, 0x80808080u, 0x80808080u, 0x80808080u};
    return v;
  };
  LAS char* xw = lds + LDS_XCH + (wm * 128 + fr) * 128 + (((wn * 4 + fq) ^ (fr & 7)) << 4);
  const LAS char* xr = lds + LDS_XCH + (wm * 128 + wn * 8 + (lane >> 3)) * 128 + (((lane & 7) ^ (lane >> 3)) << 4);
  *(LAS u32x4*)xw = quant(0);
  LGKM0(); BAR();
  store16_sys(orow, *(const LAS u32x4*)xr);
  SB();
#pragma unroll
  for (int m = 1; m < 8; ++m) *(LAS u32x4*)(xw + m * 2048) = quant(m);
  LGKM0(); BAR();
  u32x4 ov[8];
#pragma unroll
  for (int m = 1; m < 8; ++m) ov[m] = *(const LAS u32x4*)(xr + m * 2048);
#pragma unroll
  for (int m = 1; m < 8; ++m) {
    if (dst != p.rank) WAIT_V(0);
    store16_sys(orow + (size_t)m * 16 * NSH, ov[m]);
  }
  WAIT_V(0);
  BAR();
}
}

extern "C" void kernel_launch(int rank, int nranks, ncclComm_t comm, hipStream_t stream,
                              void* const* d_in, const int* in_sizes, int n_in,
                              void* d_out, int out_size, const int* in_dtypes, int out_dtype,
                              void* const* d_out_all, void* const* d_ws_all, size_t ws_bytes) {
  if (nranks != NR || (unsigned)rank >= (unsigned)NR || ws_bytes < WS_NEED || n_in != 2 || in_sizes[0] != MLOC * KD || in_sizes[1] != KD * ND ||
      out_size != NR * MLOC * NSH) { fprintf(stderr, "unsupported configuration\n"); return; }
  Params p;
  p.x = (const unsigned short*)d_in[0]; p.w = (const unsigned short*)d_in[1];
  for (int i = 0; i < NR; ++i) { p.out[i] = (char*)d_out_all[i]; p.ws[i] = (char*)d_ws_all[i]; }
  p.rank = rank; p.pad = 0;
  hipLaunchKernelGGL(gemm_a2a_q8, dim3(NWG), dim3(320), 0, stream, p);
  HIP_CHECK(hipGetLastError());
}
```
